```python
import numpy as np

kernel_name = "gpu_gemm_a2a_m4096_k4096_n16384_e5m2e5m2_dequant"


ABSMAX_RTOL = 0.05

COMPUTE_DTYPE = "float32"
EPILOGUE = "dequant"
IN_QUANT = None
OUT_QUANT = None
IN_FP8 = "e5m2e5m2"
IN_DTYPES = {"x": "e5m2", "w_mat": "e5m2",
             "scale_x": "float32", "scale_w": "float32"}
WORLD_SIZE = 8

def _snap_fp8(x, mant_bits, max_abs, min_normal):
    a = np.abs(x.astype(np.float64))
    _m, _e = np.frexp(a)
    q = np.where(a >= min_normal, np.ldexp(1.0, _e - 1 - mant_bits),
                 min_normal * 2.0 ** -mant_bits)
    snapped = np.minimum(np.round(a / q) * q, max_abs)
    return (np.sign(x) * snapped).astype(np.float32)

def setup_inputs():

    x = _snap_fp8(np.random.randn(4096, 4096).astype(np.float32), 2, 57344.0, 2.0 ** -14)
    w_mat = _snap_fp8(np.random.randn(4096, 16384).astype(np.float32), 2, 57344.0, 2.0 ** -14)
    scale_x = (0.02 * (1.0 + np.random.rand(1))).astype(np.float32)
    scale_w = (0.02 * (1.0 + np.random.rand(1))).astype(np.float32)
    return {"x": x, "w_mat": w_mat, "scale_x": scale_x, "scale_w": scale_w}

def reference(x, w_mat, scale_x, scale_w):
    acc = (x @ w_mat).astype(np.float32)
    y = (acc * (scale_x * scale_w)).astype(np.float32)
    return y
```

```cpp
#include <hip/hip_runtime.h>
#include <rccl/rccl.h>
#include <cstdio>
#include <cstdint>

#define HIP_CHECK(expr) do { hipError_t _e = (expr); if (_e != hipSuccess) { \
    fprintf(stderr, "HIP error %s at %s:%d\n", hipGetErrorString(_e), __FILE__, __LINE__); \
    return; } } while (0)

namespace {

typedef int      v8i   __attribute__((ext_vector_type(8)));
typedef int      v4i   __attribute__((ext_vector_type(4)));
typedef int      v2i   __attribute__((ext_vector_type(2)));
typedef unsigned u32x4 __attribute__((ext_vector_type(4)));
typedef float    f32x4 __attribute__((ext_vector_type(4)));
#define LAS __attribute__((address_space(3)))
#define DI __device__ __forceinline__

constexpr int NRANK = 8;
constexpr int KDIM = 4096, NFULL = 16384, NLOC = 2048, MLOC = 512;
constexpr int KSTEPS = 32;
constexpr int MT = 9;
constexpr int XS = MT * 2048;
constexpr int STG = XS + 32768;
constexpr int LDS_DEAD = 3 * STG;
constexpr int LDS_BYTES = LDS_DEAD + 16;
constexpr int NCOMM = 24;
constexpr int GRID = 256;

constexpr size_t WS_CNT = 0;
constexpr size_t WS_FLAG = 128u << 10;
constexpr size_t WS_DUMMY = 192u << 10;
constexpr size_t WS_STAMP = 512u << 10;
constexpr size_t WS_DATA = 1u << 20;
constexpr size_t XBYTES = (size_t)MLOC * KDIM;
constexpr size_t WS_NEED = WS_DATA + 2 * NRANK * XBYTES;
constexpr unsigned long long DEADLINE_TICKS = 50000000ull;

struct Params {
    const uint8_t* x;
    const uint8_t* w;
    const float* sx;
    const float* sw;
    float* out;
    uint8_t* ws[NRANK];
    int rank;
    int pad;
    unsigned order[32];
};

DI unsigned rfl(unsigned v) { return __builtin_amdgcn_readfirstlane(v); }
template <int N> DI void waitv() { asm volatile("s_waitcnt vmcnt(%0)" :: "i"(N) : "memory"); }
DI void barrier() { asm volatile("s_barrier" ::: "memory"); }
DI void ld16(u32x4& v, const uint8_t* p) { asm volatile("global_load_dwordx4 %0, %1, off" : "=v"(v) : "v"(p) : "memory"); }
DI void st16_sys(uint8_t* p, const u32x4& v) { asm volatile("global_store_dwordx4 %0, %1, off sc0 sc1\n\ts_nop 1" :: "v"(p), "v"(v) : "memory"); }
DI void st4_sys(uint8_t* p, unsigned v) { asm volatile("global_store_dword %0, %1, off sc0 sc1" :: "v"(p), "v"(v) : "memory"); }
DI __amdgpu_buffer_rsrc_t mk_srd(const void* ptr, unsigned bytes) {
    const uintptr_t a = (uintptr_t)ptr;
    const uintptr_t u = ((uintptr_t)rfl((unsigned)(a >> 32)) << 32) | rfl((unsigned)a);
    return __builtin_amdgcn_make_buffer_rsrc((void*)u, 0, (int)rfl(bytes), 0x00020000);
}
DI void dma(unsigned m0, unsigned voff, __amdgpu_buffer_rsrc_t srd, unsigned soff) {
    asm volatile("s_mov_b32 m0, %0\n\ts_nop 0\n\tbuffer_load_dwordx4 %1, %2, %3 offen lds" :: "s"(m0), "v"(voff), "s"(srd), "s"(soff) : "memory");
}
DI void dma_sys(unsigned m0, unsigned voff, __amdgpu_buffer_rsrc_t srd, unsigned soff) {
    asm volatile("s_mov_b32 m0, %0\n\ts_nop 0\n\tbuffer_load_dwordx4 %1, %2, %3 offen sc0 sc1 lds" :: "s"(m0), "v"(voff), "s"(srd), "s"(soff) : "memory");
}
#define STAMP(i) do {} while (0)
DI unsigned next_call(uint8_t* ws, int wave_global, int lane) {
    unsigned c = 0;
    if (lane == 0) c = __hip_atomic_fetch_add((unsigned*)(ws + WS_CNT + (size_t)wave_global * 64), 1u, __ATOMIC_RELAXED, __HIP_MEMORY_SCOPE_AGENT);
    return rfl(c) + 1u;
}

DI void comm_role(const Params& p, int cw  , int lane, unsigned call) {
    constexpr int NW = NCOMM * 4, NUNIT = 128 * 4 * 7;
    const int n = (NUNIT - cw + NW - 1) / NW;
    const size_t slot_off = WS_DATA + ((size_t)(call & 1u) * NRANK + p.rank) * XBYTES;
    const unsigned lrow = lane >> 3;
    const unsigned src0 = lrow * KDIM + (((lane & 7) ^ (lane >> 4)) << 4);
    const unsigned src1 = lrow * KDIM + (((lane & 7) ^ (4 + (lane >> 4))) << 4);
    u32x4 buf[2][4];
    auto decode = [&](int i, unsigned& sub, unsigned& ks, unsigned& q, unsigned& peer) {
        unsigned u = (unsigned)min(cw + i * NW, NUNIT - 1);
        unsigned oq = u / 7u; unsigned j = u - oq * 7u;
        unsigned o = oq >> 2; q = oq & 3u;
        unsigned c = (p.order[o >> 2] >> ((o & 3u) * 8u)) & 0xffu;
        sub = c & 3u; ks = c >> 2; peer = ((unsigned)p.rank + 1u + j) & 7u;
    };
    auto L = [&](int i, u32x4 (&b)[4]) {
        unsigned sub, ks, q, peer; decode(i, sub, ks, q, peer);
        const uint8_t* s = p.x + (size_t)(sub * 128 + q * 32) * KDIM + ks * 128;
        ld16(b[0], s + src0); ld16(b[1], s + 8 * KDIM + src1); ld16(b[2], s + 16 * KDIM + src0); ld16(b[3], s + 24 * KDIM + src1);
    };
    auto S = [&](int i, u32x4 (&b)[4]) {
        unsigned sub, ks, q, peer; decode(i, sub, ks, q, peer);
        uint8_t* d = p.ws[peer] + slot_off + (size_t)((sub * 32 + ks) * 16384 + q * 4096) + lane * 16;
        st16_sys(d, b[0]); st16_sys(d + 1024, b[1]); st16_sys(d + 2048, b[2]); st16_sys(d + 3072, b[3]);
    };
    auto F = [&](int i) {
        unsigned sub, ks, q, peer; decode(i < 0 ? 0 : i, sub, ks, q, peer);
        uint8_t* f = p.ws[peer] + WS_FLAG + (size_t)(((p.rank * 4 + sub) * 4 + q) * 32 + ks) * 4;
        if (i < 0) f = p.ws[p.rank] + WS_DUMMY + cw * 64;
        if (lane == 0) st4_sys(f, call);
    };
    L(0, buf[0]); L(1, buf[1]);
    waitv<0>();
    STAMP(1); waitv<0>();
    for (int i = 0; i < n; i += 2) {
        S(i, buf[0]); L(i + 2, buf[0]);
        waitv<9>();
        F(i - 1);
        if (i + 1 < n) {
            S(i + 1, buf[1]); L(i + 3, buf[1]);
            waitv<9>();
            F(i);
        }
    }
    waitv<0>();
#pragma unroll
    for (int r = 0; r < 2; ++r)
#pragma unroll
        for (int u = 0; u < 4; ++u) asm volatile("" :: "v"(buf[r][u]));
    F(n - 1);
    waitv<0>(); STAMP(2);
}

struct Tile { int rb, cb; };
DI void loader_role(const Params& p, const Tile tl, int li, int lane, unsigned call, LAS uint8_t* lds) {
    const bool remote = tl.rb < 28;
    const int src = remote ? ((p.rank + 1 + (tl.rb % 7)) & 7) : p.rank, sub = tl.rb / 7;
    uint8_t* myws = p.ws[p.rank];
    const uint8_t* xb = remote ? myws + WS_DATA + (((size_t)(call & 1u) * NRANK + src) * 4 + sub) * (XBYTES / 4) : p.x + (size_t)448 * KDIM;
    const __amdgpu_buffer_rsrc_t sX = mk_srd(xb, remote ? (unsigned)(XBYTES / 4) : 64 * KDIM);
    const __amdgpu_buffer_rsrc_t sE = mk_srd(p.x + (size_t)(remote ? tl.rb * 16 : 0) * KDIM, 16 * KDIM);
    const size_t wcol = (size_t)p.rank * NLOC + tl.cb * 256;
    const __amdgpu_buffer_rsrc_t sW = mk_srd(p.w + wcol, (unsigned)((size_t)KDIM * NFULL - wcol));
    const unsigned rowm0 = (lane >> 3) * KDIM + (((lane & 7) ^ (lane >> 4)) << 4), rowm1 = (lane >> 3) * KDIM + (((lane & 7) ^ (4 + (lane >> 4))) << 4);
    const unsigned vx0 = remote ? lane * 16 : rowm0, vx1 = remote ? lane * 16 : rowm1;
    const unsigned xk = remote ? 16384 : 128, xp = remote ? 1024 : 8 * KDIM;
    const unsigned ve = (li & 1) ? rowm1 + 8 * KDIM : rowm0;
    const unsigned vw = (lane >> 4) * NFULL + (((lane & 15) ^ ((4 * (li & 1) + (lane >> 4)) | ((li >> 1) << 3))) << 4);
    const unsigned ldsb = (unsigned)(uintptr_t)lds;
    const unsigned* fl = (const unsigned*)(myws + WS_FLAG) + ((src * 4 + sub) * 4 + li) * 32 + (lane & 31);
    auto issueX = [&](int t) {
        const unsigned sb = ldsb + (unsigned)(t % 3) * STG + li * 4096, so = t * xk + li * 4 * xp;
        dma_sys(sb, vx0, sX, so); dma_sys(sb + 1024, vx1, sX, so + xp); dma_sys(sb + 2048, vx0, sX, so + 2 * xp); dma_sys(sb + 3072, vx1, sX, so + 3 * xp);
    };
    auto issueW = [&](int t) {
        const unsigned sb = ldsb + (unsigned)(t % 3) * STG;
        dma(sb + 16384 + (li & 1) * 1024, ve, sE, t * 128);
        const unsigned p0 = (li & 1) + 4 * (li >> 1);
#pragma unroll
        for (int i = 0; i < 8; ++i) { const unsigned pc = p0 + 2 * (i & 1) + 8 * (i >> 1); dma(sb + XS + pc * 1024, vw, sW, (t * 128 + 4 * pc) * NFULL); }
    };
    int avail = remote ? 0 : KSTEPS, issued = 0; unsigned dead = 0;
    auto poll = [&]() {
        unsigned v;
        asm volatile("global_load_dword %0, %1, off sc0 sc1\n\ts_waitcnt vmcnt(0)" : "=&v"(v) : "v"(fl) : "memory");
        const unsigned m = (unsigned)__builtin_amdgcn_ballot_w64(v >= call);
        avail = (m == 0xffffffffu) ? KSTEPS : __builtin_ctz(~m);
    };
    asm volatile("s_nop 4" ::: "memory");
    issueW(0); issueW(1);
    for (int t = -1; t < KSTEPS - 1; ++t) {
        if (issued <= t + 1) {
            if (avail <= t + 1) {
                const unsigned long long t0 = __builtin_amdgcn_s_memrealtime();
                for (;;) {
                    poll();
                    if (avail > t + 1) break;
                    if (__builtin_amdgcn_s_memrealtime() - t0 > DEADLINE_TICKS) { dead = 1; avail = KSTEPS; break; }
                    __builtin_amdgcn_s_sleep(1);
                }
            }
            issueX(t + 1); issued = t + 2;
        } else if (avail <= t + 2 && t + 2 < KSTEPS) {
            poll();
        }
        const bool more = t + 2 < KSTEPS;
        if (more && avail > t + 2) {
            issueX(t + 2); issued = t + 3;
            if (t >= 0) { issueW(t + 2); waitv<13>(); } else waitv<4>();
        } else if (more && t >= 0) {
            issueW(t + 2); waitv<9>();
        } else {
            waitv<0>();
        }
        if (t == KSTEPS - 2) {
            if (lane == 0) *(LAS unsigned*)(lds + LDS_DEAD + li * 4) = dead;
            asm volatile("s_waitcnt lgkmcnt(0)" ::: "memory");
        }
        barrier();
    }
}

#define GAP() __builtin_amdgcn_sched_barrier(0)
DI v8i rd_x(LAS uint8_t* a0, LAS uint8_t* a1, int mt) {
    const v4i lo = *(LAS v4i*)(a0 + mt * 2048), hi = *(LAS v4i*)(a1 + mt * 2048);
    return __builtin_shufflevector(lo, hi, 0, 1, 2, 3, 4, 5, 6, 7);
}
DI v8i rd_w(LAS uint8_t* a) {
    const v2i r0 = __builtin_amdgcn_ds_read_tr8_b64_v2i32((LAS v2i*)(a + XS));
    const v2i r1 = __builtin_amdgcn_ds_read_tr8_b64_v2i32((LAS v2i*)(a + XS + 2048));
    const v2i r2 = __builtin_amdgcn_ds_read_tr8_b64_v2i32((LAS v2i*)(a + XS + 16384));
    const v2i r3 = __builtin_amdgcn_ds_read_tr8_b64_v2i32((LAS v2i*)(a + XS + 16384 + 2048));
    const v4i lo = __builtin_shufflevector(r0, r1, 0, 1, 2, 3), hi = __builtin_shufflevector(r2, r3, 0, 1, 2, 3);
    return __builtin_shufflevector(lo, hi, 0, 1, 2, 3, 4, 5, 6, 7);
}
#define MM(mt, nt, xf) do { acc[mt][nt] = __builtin_amdgcn_mfma_scale_f32_16x16x128_f8f6f4(W[nt], xf, acc[mt][nt], 1, 1, 0, 0, 0, 0); } while (0)

DI void mfma_role(const Params& p, const Tile tl, int w, int lane, LAS uint8_t* lds) {
    const int r = lane & 15, g = lane >> 4, q = r >> 1;
    f32x4 acc[MT][4];
#pragma unroll
    for (int m = 0; m < MT; ++m)
#pragma unroll
        for (int n = 0; n < 4; ++n) acc[m][n] = (f32x4){0.f, 0.f, 0.f, 0.f};
    v8i W[4], x7 = {}, x8 = {};
#pragma unroll
    for (int n = 0; n < 4; ++n) W[n] = (v8i){};
    const int xa0 = r * 128 + ((g ^ q) << 4), xa1 = r * 128 + (((g + 4) ^ q) << 4);
    int wa[4];
#pragma unroll
    for (int n = 0; n < 4; ++n) wa[n] = (16 * g + q) * 256 + (((4 * w + n) ^ (q | ((g & 1) << 3))) << 4) + 8 * (lane & 1);
    float sc = p.sx[0] * p.sw[0];
    asm volatile("" : "+v"(sc));
    int so = 0;
    for (int t = 0; t < KSTEPS; ++t) {
        LAS uint8_t* s = lds + so;
        LAS uint8_t* a0 = s + xa0; LAS uint8_t* a1 = s + xa1;
        so = (so == 2 * STG) ? 0 : so + STG;
        barrier(); GAP();
        const v8i x0 = rd_x(a0, a1, 0); GAP();
        MM(7, 0, x7); GAP();
        const v8i x1 = rd_x(a0, a1, 1); GAP();
        MM(8, 0, x8); GAP();
        W[0] = rd_w(s + wa[0]); GAP();
        MM(7, 1, x7); GAP();
        const v8i x2 = rd_x(a0, a1, 2); GAP();
        MM(8, 1, x8); GAP();
        W[1] = rd_w(s + wa[1]); GAP();
        MM(7, 2, x7); GAP();
        MM(8, 2, x8); GAP();
        W[2] = rd_w(s + wa[2]); GAP();
        MM(7, 3, x7); GAP();
        MM(8, 3, x8); GAP();
        W[3] = rd_w(s + wa[3]); GAP();
        MM(0, 0, x0); GAP(); MM(1, 0, x1); GAP(); MM(2, 0, x2); GAP();
        MM(0, 1, x0); GAP(); MM(1, 1, x1); GAP();
        const v8i x3 = rd_x(a0, a1, 3); GAP();
        MM(2, 1, x2); GAP();
        MM(0, 2, x0); GAP();
        const v8i x4 = rd_x(a0, a1, 4); GAP();
        MM(1, 2, x1); GAP(); MM(2, 2, x2); GAP();
        MM(0, 3, x0); GAP(); MM(1, 3, x1); GAP(); MM(2, 3, x2); GAP();
        const v8i x5 = rd_x(a0, a1, 5); GAP();
        MM(3, 0, x3); GAP(); MM(4, 0, x4); GAP();
        const v8i x6 = rd_x(a0, a1, 6); GAP();
        MM(3, 1, x3); GAP(); MM(4, 1, x4); GAP(); MM(3, 2, x3); GAP(); MM(4, 2, x4); GAP(); MM(3, 3, x3); GAP(); MM(4, 3, x4); GAP();
        x7 = rd_x(a0, a1, 7); GAP();
        MM(5, 0, x5); GAP(); MM(6, 0, x6); GAP();
        x8 = rd_x(a0, a1, 8); GAP();
        MM(5, 1, x5); GAP(); MM(6, 1, x6); GAP(); MM(5, 2, x5); GAP(); MM(6, 2, x6); GAP(); MM(5, 3, x5); GAP(); MM(6, 3, x6); GAP();
        asm volatile("s_waitcnt lgkmcnt(0)" : "+v"(x7), "+v"(x8) :: "memory"); GAP();
    }
    const u32x4 dd = *(LAS u32x4*)(lds + LDS_DEAD);
#pragma unroll
    for (int n = 0; n < 4; ++n) { MM(7, n, x7); MM(8, n, x8); }
    STAMP(1);
    if (dd.x | dd.y | dd.z | dd.w) sc = __builtin_nanf("");
    const bool remote = tl.rb < 28;
    const int src = (p.rank + 1 + (tl.rb % 7)) & 7, sub = tl.rb / 7;
    const int row_main = remote ? src * MLOC + sub * 128 : p.rank * MLOC + 448;
    const int nmain = remote ? 8 : 4;
    float* ob = p.out + (size_t)r * NLOC + tl.cb * 256 + w * 64 + 4 * g;
#pragma unroll
    for (int m = 0; m < 8; ++m) {
        if (m < nmain) {
            float* o = ob + (size_t)(row_main + 16 * m) * NLOC;
#pragma unroll
            for (int n = 0; n < 4; ++n) *(f32x4*)(o + 16 * n) = acc[m][n] * sc;
        }
    }
    if (remote) {
        float* o = ob + (size_t)(p.rank * MLOC + tl.rb * 16) * NLOC;
#pragma unroll
        for (int n = 0; n < 4; ++n) *(f32x4*)(o + 16 * n) = acc[8][n] * sc;
    }
    STAMP(2);
    STAMP(3);
}

__global__ __launch_bounds__(512, 2) void gemm_ag_kernel(const Params p) {
    extern __shared__ __attribute__((aligned(16))) uint8_t smem[];
    LAS uint8_t* lds = (LAS uint8_t*)smem;
    const int lane = threadIdx.x & 63, wave = rfl(threadIdx.x >> 6), b = blockIdx.x;
    STAMP(0);
    if (b < NCOMM) {
        if (wave < 4) {
            const unsigned call = next_call(p.ws[p.rank], b * 8 + wave, lane);
            comm_role(p, b * 4 + wave, lane, call);
        }
        return;
    }
    Tile tl;
    { const int xcd = b & 7, j = (b - NCOMM) >> 3;
      tl.rb = j; tl.cb = xcd;
    }
    if (wave >= 4) {
        const unsigned call = next_call(p.ws[p.rank], b * 8 + wave, lane);
        loader_role(p, tl, wave - 4, lane, call, lds);
    } else {
        mfma_role(p, tl, wave, lane, lds);
    }
}

}

extern "C" void kernel_launch(int rank, int nranks, ncclComm_t comm, hipStream_t stream,
                              void* const* d_in, const int* in_sizes, int n_in,
                              void* d_out, int out_size,
                              const int* in_dtypes, int out_dtype,
                              void* const* d_out_all,
                              void* const* d_ws_all, size_t ws_bytes) {
    if (nranks != NRANK || ws_bytes < WS_NEED || (unsigned)rank >= (unsigned)NRANK) { fprintf(stderr, "unexpected configuration\n"); return; }
    static bool attr_set = false;
    if (!attr_set) {
        HIP_CHECK(hipFuncSetAttribute((const void*)gemm_ag_kernel, hipFuncAttributeMaxDynamicSharedMemorySize, LDS_BYTES));
        attr_set = true;
    }
    Params p;
    p.x = (const uint8_t*)d_in[0]; p.w = (const uint8_t*)d_in[1]; p.sx = (const float*)d_in[2]; p.sw = (const float*)d_in[3];
    p.out = (float*)d_out;
    for (int r = 0; r < NRANK; ++r) p.ws[r] = (uint8_t*)d_ws_all[r];
    p.rank = rank; p.pad = 0;
    {
#define STAG 2
        uint8_t ord[128]; int n = 0;
        for (int key = 0; key < KSTEPS + 3 * STAG; ++key)
            for (int sub = 0; sub < 4; ++sub) { const int ks = key - STAG * sub; if (ks >= 0 && ks < KSTEPS) ord[n++] = (uint8_t)(sub | (ks << 2)); }
        for (int i = 0; i < 32; ++i) p.order[i] = ord[4 * i] | (ord[4 * i + 1] << 8) | (ord[4 * i + 2] << 16) | ((unsigned)ord[4 * i + 3] << 24);
    }
    hipLaunchKernelGGL(gemm_ag_kernel, dim3(GRID), dim3(512), LDS_BYTES, stream, p);
    HIP_CHECK(hipGetLastError());
}
```

```cpp
#include <hip/hip_runtime.h>
#include <rccl/rccl.h>
#include <cstdio>
#include <cstdint>
#include <type_traits>

#define HIP_CHECK(expr) do { hipError_t _e = (expr); if (_e != hipSuccess) { \
    fprintf(stderr, "HIP error %s at %s:%d\n", hipGetErrorString(_e), __FILE__, __LINE__); \
    return; } } while (0)

namespace {

typedef int      v8i   __attribute__((ext_vector_type(8)));
typedef int      v4i   __attribute__((ext_vector_type(4)));
typedef int      v2i   __attribute__((ext_vector_type(2)));
typedef unsigned u32x4 __attribute__((ext_vector_type(4)));
typedef unsigned u32x2 __attribute__((ext_vector_type(2)));
typedef float    f32x4 __attribute__((ext_vector_type(4)));
typedef float    f32x16 __attribute__((ext_vector_type(16)));
#define LAS __attribute__((address_space(3)))
#define DI __device__ __forceinline__

constexpr int NRANK = 8;
constexpr int KDIM = 4096, NFULL = 16384, NLOC = 2048, MLOC = 512;
constexpr int KSTEPS = 32;
constexpr int XS = 160 * 128;
constexpr int NHEAVY = 11;
constexpr int STG = XS + 32768;
constexpr int LDS_DEAD = 3 * STG;
constexpr int LDS_BYTES = LDS_DEAD + 16;
constexpr int NCOMM = 24;
constexpr int GRID = 256;

constexpr size_t WS_CNT = 0;
constexpr size_t WS_FLAG = 128u << 10;
constexpr size_t WS_STAMP = 512u << 10;
constexpr size_t WS_DATA = 1u << 20;
constexpr size_t XBYTES = (size_t)MLOC * KDIM;
constexpr size_t WS_NEED = WS_DATA + 2 * NRANK * XBYTES;
constexpr unsigned long long DEADLINE_TICKS = 50000000ull;

struct Params {
    const uint8_t* x;
    const uint8_t* w;
    const float* sx;
    const float* sw;
    float* out;
    uint8_t* ws[NRANK];
    int rank;
    int pad;
    unsigned order[32];
};

DI unsigned rfl(unsigned v) { return __builtin_amdgcn_readfirstlane(v); }
template <int N> DI void waitv() { asm volatile("s_waitcnt vmcnt(%0)" :: "i"(N) : "memory"); }
DI void barrier() { asm volatile("s_barrier" ::: "memory"); }
DI void st16_sys(uint8_t* p, const u32x4& v) { asm volatile("global_store_dwordx4 %0, %1, off sc0 sc1\n\ts_nop 1" :: "v"(p), "v"(v) : "memory"); }
#define OUT_AUX 16
DI void st8_sys(uint8_t* p, const u32x2& v) { asm volatile("global_store_dwordx2 %0, %1, off sc0 sc1\n\ts_nop 1" :: "v"(p), "v"(v) : "memory"); }
DI __amdgpu_buffer_rsrc_t mk_srd(const void* ptr, unsigned bytes) {
    const uintptr_t a = (uintptr_t)ptr;
    const uintptr_t u = ((uintptr_t)rfl((unsigned)(a >> 32)) << 32) | rfl((unsigned)a);
    return __builtin_amdgcn_make_buffer_rsrc((void*)u, 0, (int)rfl(bytes), 0x00020000);
}
DI void set_m0(unsigned m) { asm volatile("s_mov_b32 m0, %0\n\ts_nop 0" :: "s"(m) : "memory"); }
template <int OFF> DI void dma(unsigned voff, __amdgpu_buffer_rsrc_t srd, unsigned soff) {
    asm volatile("buffer_load_dwordx4 %0, %1, %2 offen offset:%3 lds" :: "v"(voff), "s"(srd), "s"(soff), "i"(OFF) : "memory");
}
#define GAP() __builtin_amdgcn_sched_barrier(0)
struct Stamps {};
#define STAMP(i) do {} while (0)
#define STAMP_FLUSH() do {} while (0)
#define SEG(i) do {} while (0)
DI void call_issue(unsigned& c, uint8_t* ws, int wave_global, int lane) {
    uint8_t* q = ws + WS_CNT + (size_t)wave_global * 64;
    c = 0;
    if (lane == 0) asm volatile("global_atomic_add %0, %1, %2, off sc0" : "+v"(c) : "v"(q), "v"(1u) : "memory");
}
DI unsigned call_get(unsigned& c) { asm volatile("" : "+v"(c)); return rfl(c) + 1u; }

constexpr int MAXEXC = 128;
constexpr unsigned META_RAW = 1u << 16, META_READY = 1u << 31;
constexpr int CL_META = 96 * 1024;
constexpr int CL_SCR = 100 * 1024;

template <int CTRL, int RM> DI unsigned dpp_add(unsigned v) { return v + __builtin_amdgcn_update_dpp(0u, v, CTRL, RM, 0xf, true); }
DI unsigned wave_scan(unsigned v) {
    v = dpp_add<0x111, 0xf>(v); v = dpp_add<0x112, 0xf>(v); v = dpp_add<0x114, 0xf>(v); v = dpp_add<0x118, 0xf>(v);
    v = dpp_add<0x142, 0xa>(v);
    return dpp_add<0x143, 0xc>(v);
}
DI unsigned bfi(unsigned m, unsigned a, unsigned b) { return (a & m) | (b & ~m); }

DI void encode_unit(const u32x4 (&x)[4], LAS uint8_t* slot, LAS unsigned* metaw, LAS uint8_t* scr, int lane) {
    *(LAS unsigned*)(slot + 3072 + lane * 4) = 0u;
    u32x4 t[4];
#pragma unroll
    for (int j = 0; j < 4; ++j) t[j] = x[j] & 0x7f7f7f7fu;
    unsigned sum = 0;
#pragma unroll
    for (int j = 0; j < 4; ++j) sum = __builtin_amdgcn_sad_u8(t[j][j], 0u, sum);
    const int bi = ((int)__builtin_amdgcn_readlane(wave_scan(sum), 63) - 20685) >> 10;
    const unsigned base = (unsigned)(bi < 0 ? 0 : bi > 96 ? 96 : bi);
    const unsigned k4 = (128u - base) * 0x01010101u;
    unsigned g[4];
#pragma unroll
    for (int j = 0; j < 4; ++j) {
        t[j] += k4;
        *(LAS u32x4*)(scr + j * 1024 + lane * 16) = t[j];
        const u32x4 fl = (~t[j] | (t[j] - 0x20202020u)) & 0x80808080u;
        const unsigned lo = __builtin_amdgcn_udot4(fl[1], 0x80402010u, __builtin_amdgcn_udot4(fl[0], 0x08040201u, 0u, false), false);
        const unsigned hi = __builtin_amdgcn_udot4(fl[3], 0x80402010u, __builtin_amdgcn_udot4(fl[2], 0x08040201u, 0u, false), false);
        g[j] = (hi << 8) + lo;
    }
    unsigned m0 = (g[0] >> 7) | (g[1] << 9), m1 = (g[2] >> 7) | (g[3] << 9);
    const unsigned c0 = __builtin_popcount(m0), c = c0 + __builtin_popcount(m1);
    const unsigned incl = wave_scan(c), cnt = __builtin_amdgcn_readlane(incl, 63);
    unsigned meta;
    if (cnt <= MAXEXC) {
        LAS uint8_t* a0 = slot + 3072 + (incl - c) * 2; LAS uint8_t* a1 = a0 + c0 * 2;
        LAS uint8_t* dump = scr + 4096;
        const unsigned l16 = lane * 16;
        auto round = [&]() {
            const unsigned b0 = __builtin_ctz(m0 | 0x80000000u), b1 = __builtin_ctz(m1 | 0x80000000u);
            *(LAS unsigned short*)(m0 ? a0 : dump) = (unsigned short)(l16 + (b0 & 15u) + ((b0 & 16u) << 6));
            *(LAS unsigned short*)(m1 ? a1 : dump) = (unsigned short)(l16 + 2048u + (b1 & 15u) + ((b1 & 16u) << 6));
            m0 &= m0 - 1u; m1 &= m1 - 1u; a0 += 2; a1 += 2;
        };
        round(); round(); round();
        while (__builtin_amdgcn_ballot_w64((m0 | m1) != 0u)) round();
        const unsigned e = *(LAS unsigned*)(slot + 3072 + lane * 4);
        const unsigned tl = scr[e & 0xfffu], th = scr[e >> 16];
        u32x4 a, b, cc;
#pragma unroll
        for (int d = 0; d < 4; ++d) {
            const unsigned td = t[3][d];
            a[d] = bfi(0x60606060u, td << 5, bfi(0x1f1f1f1fu, t[0][d], x[0][d]));
            b[d] = bfi(0x60606060u, td << 3, bfi(0x1f1f1f1fu, t[1][d], x[1][d]));
            cc[d] = bfi(0x60606060u, bfi(0x20202020u, td << 1, x[3][d] >> 1), bfi(0x1f1f1f1fu, t[2][d], x[2][d]));
        }
        *(LAS u32x4*)(slot + l16) = a; *(LAS u32x4*)(slot + 1024 + l16) = b; *(LAS u32x4*)(slot + 2048 + l16) = cc;
        const unsigned ql = (unsigned)lane * 2u < cnt ? ((tl ^ 0x80u) >> 5) << 13 : 0u, qh = (unsigned)lane * 2u + 1u < cnt ? ((th ^ 0x80u) >> 5) << 29 : 0u;
        *(LAS unsigned*)(slot + 3072 + lane * 4) = e | ql | qh;
        meta = base | (cnt << 8);
    } else {
#pragma unroll
        for (int j = 0; j < 4; ++j) *(LAS u32x4*)(slot + j * 1024 + lane * 16) = x[j];
        meta = META_RAW;
    }
    asm volatile("s_waitcnt lgkmcnt(0)" ::: "memory");
    if (lane == 0) *(volatile LAS unsigned*)metaw = meta | META_READY;
}

#define PACE_NUM 2643
DI void comm_role(const Params& p, int wave, int lane, LAS uint8_t* lds, Stamps& st) {
    const int b = blockIdx.x, n = (512 - b + NCOMM - 1) / NCOMM;
    const unsigned src0 = (lane >> 3) * KDIM + (((lane & 7) ^ (lane >> 4)) << 4);
    const unsigned src1 = (lane >> 3) * KDIM + (((lane & 7) ^ (4 + (lane >> 4))) << 4);
    auto where = [&](int i, unsigned& sub, unsigned& ks, unsigned& q) {
        const unsigned u = b + i * NCOMM, o = u >> 2; q = u & 3u;
        const unsigned c = (p.order[o >> 2] >> ((o & 3u) * 8u)) & 0xffu;
        sub = c & 3u; ks = c >> 2;
    };
    auto load = [&](int i, u32x4 (&x)[4]) {
        unsigned sub, ks, q; where(i, sub, ks, q);
        const uint8_t* s = p.x + (size_t)(sub * 128 + q * 32) * KDIM + ks * 128;
        x[0] = *(const u32x4*)(s + src0); x[1] = *(const u32x4*)(s + 8 * KDIM + src1); x[2] = *(const u32x4*)(s + 16 * KDIM + src0); x[3] = *(const u32x4*)(s + 24 * KDIM + src1);
    };
    LAS unsigned* metaw = (LAS unsigned*)(lds + CL_META);
    LAS uint8_t* scr = lds + CL_SCR + wave * 4160;
    unsigned craw = 0;
    if (wave != 7) call_issue(craw, p.ws[p.rank], b * 8 + wave, lane);
    u32x4 x0[4], xe[4], xf[4], xg[4];
    load(0, x0); load(1 + wave, xe); load(9 + wave, xf); load(min(17 + wave, n - 1), xg);
    if (wave == 0 && lane < 32) metaw[lane] = 0u;
    asm volatile("s_waitcnt lgkmcnt(0)" ::: "memory");
    barrier();
    if (wave == 7) {
        for (int i = 8; i <= 16; i += 8) {
            encode_unit(xe, lds + i * 4096, metaw + i, scr, lane);
#pragma unroll
            for (int j = 0; j < 4; ++j) xe[j] = xf[j];
        }
        return;
    }
    asm volatile("s_waitcnt vmcnt(0)" : "+v"(x0[0]), "+v"(x0[1]), "+v"(x0[2]), "+v"(x0[3]), "+v"(xe[0]), "+v"(xe[1]), "+v"(xe[2]), "+v"(xe[3]),
                                        "+v"(xf[0]), "+v"(xf[1]), "+v"(xf[2]), "+v"(xf[3]), "+v"(xg[0]), "+v"(xg[1]), "+v"(xg[2]), "+v"(xg[3]) :: "memory");
    const unsigned call = call_get(craw);
    const unsigned peer = ((unsigned)p.rank + 1u + (unsigned)wave) & 7u;
    uint8_t* const pw = p.ws[peer];
    uint8_t* const dbase = pw + WS_DATA + ((size_t)(call & 1u) * NRANK + p.rank) * XBYTES + lane * 16;
    uint8_t* const fbase = pw + WS_FLAG + (size_t)((call & 1u) * NRANK + p.rank) * 4096;
    const unsigned long long t0 = __builtin_amdgcn_s_memrealtime();
    STAMP(1);
    unsigned due = (unsigned)(b * (4160 / NCOMM) * PACE_NUM);
    for (int i = 0; i < n; ++i) {
        unsigned meta = META_RAW;
        if (i > 0) {
            for (;;) {
                meta = rfl(*(volatile LAS unsigned*)(metaw + i));
                if (meta & META_READY) break;
                __builtin_amdgcn_s_sleep(1);
            }
            asm volatile("" ::: "memory");
#pragma unroll
            for (int j = 0; j < 4; ++j) x0[j] = *(LAS u32x4*)(lds + i * 4096 + j * 1024 + lane * 16);
        }
        unsigned sub, ks, q; where(i, sub, ks, q);
        const unsigned nl = (meta & META_RAW) ? 64u : (((meta >> 8) & 0xffu) + 7u) >> 3;
        uint8_t* d = dbase + (size_t)((sub * 32 + ks) * 16384 + q * 4096);
        while ((unsigned)(__builtin_amdgcn_s_memrealtime() - t0) < (due >> 16)) __builtin_amdgcn_s_sleep(1);
        st16_sys(d, x0[0]); st16_sys(d + 1024, x0[1]); st16_sys(d + 2048, x0[2]);
        if ((unsigned)lane < nl) st16_sys(d + 3072, x0[3]);
        due += (3072u + 64u + ((nl + 3u) >> 2) * 64u) * PACE_NUM;
        const int ie = 1 + wave + 8 * i;
        if (ie < n) {
            encode_unit(xe, lds + ie * 4096, metaw + ie, scr, lane);
            if (i == 0) STAMP(3);
#pragma unroll
            for (int j = 0; j < 4; ++j) { xe[j] = xf[j]; xf[j] = xg[j]; }
        }
        waitv<0>();
        if (lane == 0) st8_sys(fbase + ((sub * 4 + q) * 32 + ks) * 8, u32x2{call, meta});
    }
}

struct Tile { int rb, cb; };
struct XRegs { u32x4 r[4]; unsigned e; };
DI void unit_load(XRegs& R, unsigned meta, const unsigned (&vx)[4], unsigned ve, __amdgpu_buffer_rsrc_t srd, unsigned so) {
    asm volatile("buffer_load_dwordx4 %0, %5, %10, %11 offen sc0 sc1\n\t"
                 "buffer_load_dwordx4 %1, %6, %10, %11 offen sc0 sc1\n\t"
                 "s_bitcmp1_b32 %12, 16\n\t"
                 "buffer_load_dwordx4 %2, %7, %10, %11 offen sc0 sc1\n\t"
                 "s_cbranch_scc0 1f\n\t"
                 "buffer_load_dwordx4 %3, %8, %10, %11 offen sc0 sc1\n\t"
                 "s_branch 2f\n"
                 "1:\n\t"
                 "buffer_load_sshort %4, %9, %10, %11 offen sc0 sc1\n"
                 "2:"
                 : "+v"(R.r[0]), "+v"(R.r[1]), "+v"(R.r[2]), "+v"(R.r[3]), "+v"(R.e)
                 : "v"(vx[0]), "v"(vx[1]), "v"(vx[2]), "v"(vx[3]), "v"(ve), "s"(srd), "s"(so), "s"(meta) : "memory", "scc");
}
DI void loader_role(const Params& p, const Tile tl, int li, int lane, LAS uint8_t* lds, Stamps& st) {
#define LOADER_PRIO 3
    __builtin_amdgcn_s_setprio(LOADER_PRIO);
    unsigned craw; call_issue(craw, p.ws[p.rank], blockIdx.x * 8 + 4 + li, lane);
    const bool remote = tl.rb < 28;
    const int src = remote ? ((p.rank + 1 + (tl.rb % 7)) & 7) : p.rank, sub = remote ? tl.rb / 7 : 0;
    uint8_t* myws = p.ws[p.rank];
    const __amdgpu_buffer_rsrc_t sE = mk_srd(p.x + (size_t)(tl.rb < NHEAVY ? 160 + 32 * tl.rb : 128) * KDIM, 32 * KDIM);
    const size_t wcol = (size_t)p.rank * NLOC + tl.cb * 256;
    const __amdgpu_buffer_rsrc_t sW = mk_srd(p.w + wcol, (unsigned)((size_t)KDIM * NFULL - wcol));
    const unsigned rowm0 = (lane >> 3) * KDIM + (((lane & 7) ^ (lane >> 4)) << 4), rowm1 = (lane >> 3) * KDIM + (((lane & 7) ^ (4 + (lane >> 4))) << 4);
    const unsigned ve = li * 8 * KDIM + ((li & 1) ? rowm1 : rowm0);
    unsigned vx[4], vw[8];
#pragma unroll
    for (int j = 0; j < 4; ++j) vx[j] = remote ? li * 4096 + j * 1024 + lane * 16 : (li * 4 + j) * 8 * KDIM + ((j & 1) ? rowm1 : rowm0);
    const unsigned vxe = li * 4096 + 3072 + lane * 2;
    const unsigned xk = remote ? 16384 : 128;
#pragma unroll
    for (int i = 0; i < 8; ++i) {
        const unsigned k = 32 * li + 4 * i + (lane >> 4);
        vw[i] = k * NFULL + (((lane & 15) ^ ((k & 7) << 1)) << 4) - (i & 3) * 1024;
    }
    const unsigned ldsb = (unsigned)(uintptr_t)lds;
    const unsigned mE = ldsb + 16384 + li * 1024, mW = ldsb + XS + li * 8192;
#define WPIECE(i, t, so) do { if constexpr ((i) == 0) { set_m0(mE + (so)); dma<0>(ve, sE, (t) * 128); } \
        else { if constexpr ((i) == 1 || (i) == 5) set_m0(mW + (so) + ((i) - 1) * 1024); dma<(((i) - 1) & 3) * 1024>(vw[(i) - 1], sW, (t) * (128 * NFULL)); } } while (0)
    auto issueW = [&](int t, unsigned so) {
        WPIECE(0, t, so); WPIECE(1, t, so); WPIECE(2, t, so); WPIECE(3, t, so); WPIECE(4, t, so); WPIECE(5, t, so); WPIECE(6, t, so); WPIECE(7, t, so); WPIECE(8, t, so);
    };
    asm volatile("s_nop 4" ::: "memory");
    issueW(0, 0);
    waitv<9>();
    const unsigned call = call_get(craw);
    const uint8_t* xb = remote ? myws + WS_DATA + (((size_t)(call & 1u) * NRANK + src) * 4 + sub) * (XBYTES / 4) : p.x;
    const __amdgpu_buffer_rsrc_t sX = mk_srd(xb, (unsigned)(XBYTES / 4));
    const uint8_t* fl = myws + WS_FLAG + (size_t)((call & 1u) * NRANK + src) * 4096 + ((sub * 4 + li) * 32 + (lane & 31)) * 8;
    asm volatile("s_nop 4" ::: "memory");

    int avail = remote ? 0 : 64; unsigned dead = 0;
    unsigned metav = META_RAW;
    u32x2 pv = {0u, 0u};
    XRegs R0 = {}, R1 = {};
    auto poll_async = [&](int lim) {
        asm volatile("s_cmp_gt_i32 %2, %3\n\ts_cbranch_scc1 1f\n\tglobal_load_dwordx2 %0, %1, off sc0 sc1\n1:" : "+v"(pv) : "v"(fl), "s"(avail), "s"(lim) : "memory", "scc");
    };
    auto take = [&](const u32x2& v) {
        const unsigned long long m = __builtin_amdgcn_ballot_w64(v.x < call);
        avail = m ? __builtin_ctzll(m) : 64;
        metav = v.y;
    };
    auto need = [&](int s) {
        asm volatile("s_cmp_gt_i32 %1, %2\n\ts_cbranch_scc1 1f\n\ts_waitcnt vmcnt(0)\n1:" : "+v"(pv) : "s"(avail), "s"(s) : "memory", "scc");
        if (avail > s) return;
        take(pv);
        if (avail <= s) {
            const unsigned long long t0 = __builtin_amdgcn_s_memrealtime();
            do {
                u32x2 v;
                asm volatile("global_load_dwordx2 %0, %1, off sc0 sc1\n\ts_waitcnt vmcnt(0)" : "=&v"(v) : "v"(fl) : "memory");
                take(v);
                if (__builtin_amdgcn_s_memrealtime() - t0 > DEADLINE_TICKS && avail <= s) { dead = 1; avail = 64; metav = META_RAW; }
            } while (avail <= s);
        }
    };
    unsigned m60 = 0x60606060u; asm("" : "+s"(m60));
    auto decode = [&](auto withw, int s, const XRegs& R, unsigned meta, unsigned so, unsigned sn) {
        constexpr bool WW = decltype(withw)::value;
        LAS uint8_t* img = lds + so + li * 4096;
        if (__builtin_expect(meta & META_RAW, 0)) {
            if constexpr (WW) { WPIECE(0, s + 1, sn); WPIECE(1, s + 1, sn); }
            *(LAS u32x4*)(img + lane * 16) = R.r[0];
            if constexpr (WW) { WPIECE(2, s + 1, sn); WPIECE(3, s + 1, sn); }
            *(LAS u32x4*)(img + 1024 + lane * 16) = R.r[1];
            if constexpr (WW) { WPIECE(4, s + 1, sn); WPIECE(5, s + 1, sn); }
            *(LAS u32x4*)(img + 2048 + lane * 16) = R.r[2];
            if constexpr (WW) { WPIECE(6, s + 1, sn); WPIECE(7, s + 1, sn); }
            *(LAS u32x4*)(img + 3072 + lane * 16) = R.r[3];
            if constexpr (WW) WPIECE(8, s + 1, sn);
        } else {
            const unsigned base4 = (meta & 0xffu) * 0x01010101u, tab = base4 + 0x90801000u;
            const unsigned nx = (meta >> 8) & 0xffu;
            auto patch = [&](unsigned e) {
                __hip_atomic_fetch_add((LAS unsigned*)(img + (e & 0xffcu)), (unsigned)((int)e >> 13) << (((e & 3u) << 3) + 5u), __ATOMIC_RELAXED, __HIP_MEMORY_SCOPE_WORKGROUP);
            };
            u32x4 sA, sB, sC;
            sA = R.r[0] & m60; asm("" : "+v"(sA));
            *(LAS u32x4*)(img + lane * 16) = (R.r[0] ^ sA) + base4;
            if constexpr (WW) { GAP(); WPIECE(0, s + 1, sn); GAP(); }
            sB = R.r[1] & m60; asm("" : "+v"(sB));
            *(LAS u32x4*)(img + 1024 + lane * 16) = (R.r[1] ^ sB) + base4;
            if constexpr (WW) { GAP(); WPIECE(1, s + 1, sn); GAP(); }
            sC = R.r[2] & m60; asm("" : "+v"(sC));
            *(LAS u32x4*)(img + 2048 + lane * 16) = (R.r[2] ^ sC) + base4;
            if constexpr (WW) { GAP(); WPIECE(2, s + 1, sn); GAP(); }
            u32x4 o3;
            typedef unsigned long long u64;
#pragma unroll
            for (int d = 0; d < 4; d += 2) {
                u64 a = (((u64)sA[d + 1] << 32) | sA[d]) >> 5, b2 = (((u64)sB[d + 1] << 32) | sB[d]) >> 3, c = (((u64)sC[d + 1] << 32) | sC[d]) >> 5;
                asm("" : "+v"(a), "+v"(b2), "+v"(c));
                o3[d] = __builtin_amdgcn_perm(tab, tab, (unsigned)c) + (unsigned)a + (unsigned)b2;
                o3[d + 1] = __builtin_amdgcn_perm(tab, tab, (unsigned)(c >> 32)) + (unsigned)(a >> 32) + (unsigned)(b2 >> 32);
                if constexpr (WW) { if (d == 0) { GAP(); WPIECE(3, s + 1, sn); GAP(); } }
            }
            *(LAS u32x4*)(img + 3072 + lane * 16) = o3;
            if constexpr (WW) { GAP(); WPIECE(4, s + 1, sn); GAP(); }
            patch((unsigned)lane < nx ? R.e : (unsigned)lane * 4u);
            if (__builtin_expect(nx > 64u, 0)) {
                unsigned e2;
                asm volatile("buffer_load_sshort %0, %1, %2, %3 offen offset:128 sc0 sc1\n\ts_waitcnt vmcnt(0)" : "=&v"(e2) : "v"(vxe), "s"(sX), "s"(s * xk) : "memory");
                patch((unsigned)lane + 64u < nx ? e2 : (unsigned)lane * 4u);
            }
            if constexpr (WW) { GAP(); WPIECE(5, s + 1, sn); WPIECE(6, s + 1, sn); WPIECE(7, s + 1, sn); WPIECE(8, s + 1, sn); }
        }
    };
    unsigned so = 0, mc, mn;
    poll_async(0);
    need(0);
    mc = __builtin_amdgcn_readlane(metav, 0);
    unit_load(R0, mc, vx, vxe, sX, 0);
    poll_async(1);
    waitv<0>();
    auto step = [&](int s, XRegs& Rc, XRegs& Rn) {
        const unsigned sn = so == 2 * STG ? 0 : so + STG;
        need(s + 1);
        SEG(1);
        mn = __builtin_amdgcn_readlane(metav, s + 1);
        unit_load(Rn, mn, vx, vxe, sX, (s + 1) * xk);
        SEG(2);
        asm volatile("s_waitcnt vmcnt(13)" : "+v"(Rc.r[0]), "+v"(Rc.r[1]), "+v"(Rc.r[2]), "+v"(Rc.r[3]), "+v"(Rc.e) :: "memory");
        SEG(3);
        decode(std::true_type{}, s, Rc, mc, so, sn);
        SEG(4);
        waitv<13>();
        poll_async(s + 2);
        so = sn; mc = mn;
        asm volatile("s_waitcnt lgkmcnt(0)" ::: "memory");
        SEG(5);
        barrier();
        SEG(6);
    };
    for (int s = 0; s < KSTEPS - 2; s += 2) { step(s, R0, R1); step(s + 1, R1, R0); }
    step(KSTEPS - 2, R0, R1);
    waitv<9>();
    asm volatile("" : "+v"(R1.r[0]), "+v"(R1.r[1]), "+v"(R1.r[2]), "+v"(R1.r[3]), "+v"(R1.e));
    decode(std::false_type{}, KSTEPS - 1, R1, mc, so, 0);
    if (lane == 0) *(LAS unsigned*)(lds + LDS_DEAD + li * 4) = dead;
    asm volatile("s_waitcnt vmcnt(0) lgkmcnt(0)" : "+v"(pv) :: "memory");
    barrier();
    STAMP(5);
    STAMP_FLUSH();
}

DI v8i rd_x(LAS uint8_t* a0, LAS uint8_t* a1, int mt) {
    const v4i lo = *(LAS v4i*)(a0 + mt * 4096), hi = *(LAS v4i*)(a1 + mt * 4096);
    return __builtin_shufflevector(lo, hi, 0, 1, 2, 3, 4, 5, 6, 7);
}
DI v8i rd_w(LAS uint8_t* a, int kh) {
    const v2i r0 = __builtin_amdgcn_ds_read_tr8_b64_v2i32((LAS v2i*)(a + XS + kh * 16384));
    const v2i r1 = __builtin_amdgcn_ds_read_tr8_b64_v2i32((LAS v2i*)(a + XS + kh * 16384 + 2048));
    const v2i r2 = __builtin_amdgcn_ds_read_tr8_b64_v2i32((LAS v2i*)(a + XS + kh * 16384 + 4096));
    const v2i r3 = __builtin_amdgcn_ds_read_tr8_b64_v2i32((LAS v2i*)(a + XS + kh * 16384 + 6144));
    const v4i lo = __builtin_shufflevector(r0, r1, 0, 1, 2, 3), hi = __builtin_shufflevector(r2, r3, 0, 1, 2, 3);
    return __builtin_shufflevector(lo, hi, 0, 1, 2, 3, 4, 5, 6, 7);
}
#define MM(mt, nt, kh, xf) do { acc[mt][nt] = __builtin_amdgcn_mfma_scale_f32_32x32x64_f8f6f4(W[nt][kh], xf, acc[mt][nt], 1, 1, 0, 0, 0, 0); GAP(); } while (0)

template <int NM>
DI void mfma_role(const Params& p, const Tile tl, int w, int lane, LAS uint8_t* lds, Stamps& st) {
    const int r = lane & 31, g = lane >> 5, q = (lane & 15) >> 1;
    f32x16 acc[NM][2];
#pragma unroll
    for (int m = 0; m < NM; ++m) { acc[m][0] = (f32x16){}; acc[m][1] = (f32x16){}; }
    v8i W[2][2], xp0 = {}, xp1 = {};
    W[0][1] = (v8i){}; W[1][1] = (v8i){};
    const int xa = r * 128 + (((2 * g) ^ ((r >> 1) & 7)) << 4);
    const int wa = (32 * g + q) * 256 + (((4 * w + ((lane >> 4) & 1)) ^ (q << 1)) << 4) + 8 * (lane & 1);
    float sc = p.sx[0] * p.sw[0];
    asm volatile("" : "+v"(sc));
    unsigned row_main = tl.rb < 28 ? ((p.rank + 1 + (tl.rb % 7)) & 7) * MLOC + (tl.rb / 7) * 128 : p.rank * MLOC;
    unsigned row_extra = p.rank * MLOC + (tl.rb < NHEAVY ? 160 + 32 * tl.rb : 128);
    __amdgpu_buffer_rsrc_t sO = mk_srd(p.out, 4096u * NLOC * 4u);
    asm volatile("" : "+s"(row_main), "+s"(row_extra), "+s"(sO));
    int so = 0;
    for (int t = 0; t < KSTEPS; ++t) {
        LAS uint8_t* a00 = lds + (so + xa); LAS uint8_t* a01 = lds + (so + (xa ^ 16)); LAS uint8_t* a10 = lds + (so + (xa ^ 64)); LAS uint8_t* a11 = lds + (so + (xa ^ 80));
        LAS uint8_t* w0 = lds + (so + wa); LAS uint8_t* w1 = lds + (so + (wa ^ 32));
        asm volatile("" : "+v"(a00), "+v"(a01), "+v"(a10), "+v"(a11), "+v"(w0), "+v"(w1));
        so = (so == 2 * STG) ? 0 : so + STG;
        barrier(); GAP();
        v8i xf[2 * NM];
        MM(NM - 2, 0, 1, xp0);
        xf[0] = rd_x(a00, a01, 0); GAP();
        W[0][0] = rd_w(w0, 0); GAP();
        MM(NM - 2, 1, 1, xp0);
        W[1][0] = rd_w(w1, 0); GAP();
        MM(NM - 1, 0, 1, xp1);
        xf[1] = rd_x(a00, a01, 1); GAP();
        MM(NM - 1, 1, 1, xp1);
#pragma unroll
        for (int f = 0; f < 2 * NM - 2; ++f) {
            const int fn = f + 2;
            xf[fn] = fn < NM ? rd_x(a00, a01, fn) : rd_x(a10, a11, fn - NM); GAP();
            MM(f % NM, 0, f / NM, xf[f]);
            if (f == 0) { W[0][1] = rd_w(w0, 1); GAP(); }
            if (f == 1) { W[1][1] = rd_w(w1, 1); GAP(); }
            MM(f % NM, 1, f / NM, xf[f]);
        }
        xp0 = xf[2 * NM - 2]; xp1 = xf[2 * NM - 1];
        asm volatile("s_waitcnt lgkmcnt(0)" : "+v"(xp0), "+v"(xp1) :: "memory"); GAP();
    }
    const u32x4 dd = *(LAS u32x4*)(lds + LDS_DEAD);
    MM(NM - 2, 0, 1, xp0); MM(NM - 2, 1, 1, xp0); MM(NM - 1, 0, 1, xp1); MM(NM - 1, 1, 1, xp1);
    STAMP(1);
    if (dd.x | dd.y | dd.z | dd.w) sc = __builtin_nanf("");
    LAS uint8_t* eb = lds + 2 * STG + w * 12288;
    const int ew = r * 128 + ((g ^ (r & 7)) << 4), er = (lane >> 3) * 128 + (((lane & 7) ^ (lane >> 3)) << 4);
    const unsigned vo = ((lane >> 3) * NLOC + tl.cb * 256 + w * 64 + 4 * (lane & 7)) * 4;
    auto stage = [&](int i, u32x4 (&v)[4]) {
        LAS uint8_t* e = eb + (i % 3) * 4096;
        const f32x16 a = acc[i >> 1][i & 1] * sc;
#pragma unroll
        for (int k = 0; k < 4; ++k) *(LAS f32x4*)(e + (ew ^ (k << 5))) = (f32x4){a[4 * k], a[4 * k + 1], a[4 * k + 2], a[4 * k + 3]};
#pragma unroll
        for (int k = 0; k < 4; ++k) v[k] = *(LAS u32x4*)(e + er + k * 1024);
    };
    auto put = [&](int i, u32x4 (&v)[4]) {
        const unsigned row = (i >> 1) < 4 ? row_main + 32 * (i >> 1) : row_extra;
#pragma unroll
        for (int k = 0; k < 4; ++k) __builtin_amdgcn_raw_buffer_store_b128(v[k], sO, vo, (row + 8 * k) * (NLOC * 4) + (i & 1) * 128, OUT_AUX);
    };
    u32x4 ev[2][4];
    stage(0, ev[0]);
#pragma unroll
    for (int i = 1; i < 2 * NM; ++i) { stage(i, ev[i & 1]); GAP(); put(i - 1, ev[(i - 1) & 1]); GAP(); }
    put(2 * NM - 1, ev[1]);
    STAMP(2);
    STAMP(3); STAMP_FLUSH();
}

__global__ __launch_bounds__(512, 2) void gemm_ag_kernel(const Params p) {
    extern __shared__ __attribute__((aligned(16))) uint8_t smem[];
    LAS uint8_t* lds = (LAS uint8_t*)smem;
    const int lane = threadIdx.x & 63, wave = rfl(threadIdx.x >> 6), b = blockIdx.x;
    Stamps st = {};
    STAMP(0);
    if (b < NCOMM) {
        comm_role(p, wave, lane, lds, st);
        return;
    }
    Tile tl;
    { const int xcd = b & 7, j = (b - NCOMM) >> 3;
      tl.rb = j; tl.cb = xcd;
    }
    if (wave >= 4) {
        loader_role(p, tl, wave - 4, lane, lds, st);
    } else {
        if (tl.rb < NHEAVY || tl.rb == 28) mfma_role<5>(p, tl, wave, lane, lds, st);
        else mfma_role<4>(p, tl, wave, lane, lds, st);
    }
}

}

extern "C" void kernel_launch(int rank, int nranks, ncclComm_t comm, hipStream_t stream,
                              void* const* d_in, const int* in_sizes, int n_in,
                              void* d_out, int out_size,
                              const int* in_dtypes, int out_dtype,
                              void* const* d_out_all,
                              void* const* d_ws_all, size_t ws_bytes) {
    if (nranks != NRANK || ws_bytes < WS_NEED || (unsigned)rank >= (unsigned)NRANK) { fprintf(stderr, "unexpected configuration\n"); return; }
    static bool attr_set = false;
    if (!attr_set) {
        HIP_CHECK(hipFuncSetAttribute((const void*)gemm_ag_kernel, hipFuncAttributeMaxDynamicSharedMemorySize, LDS_BYTES));
        attr_set = true;
    }
    Params p;
    p.x = (const uint8_t*)d_in[0]; p.w = (const uint8_t*)d_in[1]; p.sx = (const float*)d_in[2]; p.sw = (const float*)d_in[3];
    p.out = (float*)d_out;
    for (int r = 0; r < NRANK; ++r) p.ws[r] = (uint8_t*)d_ws_all[r];
    p.rank = rank; p.pad = 0;
    {
#define STAG 0
        uint8_t ord[128]; int n = 0;
        for (int key = 0; key < KSTEPS + 3 * STAG; ++key)
            for (int sub = 0; sub < 4; ++sub) { const int ks = key - STAG * sub; if (ks >= 0 && ks < KSTEPS) ord[n++] = (uint8_t)(sub | (ks << 2)); }
        for (int i = 0; i < 32; ++i) p.order[i] = ord[4 * i] | (ord[4 * i + 1] << 8) | (ord[4 * i + 2] << 16) | ((unsigned)ord[4 * i + 3] << 24);
    }
    hipLaunchKernelGGL(gemm_ag_kernel, dim3(GRID), dim3(512), LDS_BYTES, stream, p);
    HIP_CHECK(hipGetLastError());
}
```

```cpp
#include <hip/hip_runtime.h>
#include <rccl/rccl.h>
#include <cstdio>
#include <cstdint>
#include <type_traits>

#define HIP_CHECK(expr) do { hipError_t _e = (expr); if (_e != hipSuccess) { \
    fprintf(stderr, "HIP error %s at %s:%d\n", hipGetErrorString(_e), __FILE__, __LINE__); \
    return; } } while (0)

namespace {

typedef int      v8i   __attribute__((ext_vector_type(8)));
typedef int      v4i   __attribute__((ext_vector_type(4)));
typedef int      v2i   __attribute__((ext_vector_type(2)));
typedef unsigned u32x4 __attribute__((ext_vector_type(4)));
typedef unsigned u32x2 __attribute__((ext_vector_type(2)));
typedef float    f32x4 __attribute__((ext_vector_type(4)));
typedef float    f32x16 __attribute__((ext_vector_type(16)));
#define LAS __attribute__((address_space(3)))
#define DI __device__ __forceinline__

constexpr int NRANK = 8;
constexpr int KDIM = 4096, NFULL = 16384, NLOC = 2048, MLOC = 512;
constexpr int KSTEPS = 32;
constexpr int XS = 160 * 128;
constexpr int NHEAVY = 11;
constexpr int STG = XS + 32768;
constexpr int LDS_DEAD = 3 * STG;
constexpr int LDS_BYTES = LDS_DEAD + 16;
constexpr int NCOMM = 24;
constexpr int GRID = 256;

constexpr size_t WS_CNT = 0;
constexpr size_t WS_FLAG = 128u << 10;
constexpr size_t WS_STAMP = 512u << 10;
constexpr size_t WS_DATA = 2u << 20;
constexpr size_t XBYTES = (size_t)MLOC * KDIM;
constexpr size_t WS_NEED = WS_DATA + 2 * NRANK * XBYTES;
constexpr unsigned long long DEADLINE_TICKS = 50000000ull;

struct Params {
    const uint8_t* x;
    uint8_t* myws;
    const uint8_t* w;
    int rank;
    int pad;
    unsigned order[32];
    uint8_t* peer[NRANK];
    const float* sx;
    const float* sw;
    float* out;
};

DI unsigned rfl(unsigned v) { return __builtin_amdgcn_readfirstlane(v); }
template <int N> DI void waitv() { asm volatile("s_waitcnt vmcnt(%0)" :: "i"(N) : "memory"); }
DI void barrier() { asm volatile("s_barrier" ::: "memory"); }
#define OUT_AUX 16
DI void st8_sys(uint8_t* p, const u32x2& v) { asm volatile("global_store_dwordx2 %0, %1, off sc0 sc1\n\ts_nop 1" :: "v"(p), "v"(v) : "memory"); }
DI __amdgpu_buffer_rsrc_t mk_srd(const void* ptr, unsigned bytes) {
    const uintptr_t a = (uintptr_t)ptr;
    const uintptr_t u = ((uintptr_t)rfl((unsigned)(a >> 32)) << 32) | rfl((unsigned)a);
    return __builtin_amdgcn_make_buffer_rsrc((void*)u, 0, (int)rfl(bytes), 0x00020000);
}
DI void set_m0(unsigned m) { asm volatile("s_mov_b32 m0, %0\n\ts_nop 0" :: "s"(m) : "memory"); }
template <int OFF> DI void dma(unsigned voff, __amdgpu_buffer_rsrc_t srd, unsigned soff) {
    asm volatile("buffer_load_dwordx4 %0, %1, %2 offen offset:%3 lds" :: "v"(voff), "s"(srd), "s"(soff), "i"(OFF) : "memory");
}
#define GAP() __builtin_amdgcn_sched_barrier(0)
struct Stamps {};
#define STAMP(i) do {} while (0)
#define STAMP_FLUSH() do {} while (0)
#define SEG(i) do {} while (0)
DI void call_issue(unsigned& c, uint8_t* ws, int wave_global, int lane) {
    uint8_t* q = ws + WS_CNT + (size_t)wave_global * 64;
    c = 0;
    if (lane == 0) asm volatile("global_atomic_add %0, %1, %2, off sc0" : "+v"(c) : "v"(q), "v"(1u) : "memory");
}
DI unsigned call_get(unsigned& c) { asm volatile("" : "+v"(c)); return rfl(c) + 1u; }

constexpr int MAXEXC = 128;
constexpr unsigned META_RAW = 1u << 16, META_READY = 1u << 31;
constexpr int CL_META = 96 * 1024;
constexpr int CL_SCR = 100 * 1024;

template <int CTRL, int RM> DI unsigned dpp_add(unsigned v) { return v + __builtin_amdgcn_update_dpp(0u, v, CTRL, RM, 0xf, true); }
DI unsigned wave_scan(unsigned v) {
    v = dpp_add<0x111, 0xf>(v); v = dpp_add<0x112, 0xf>(v); v = dpp_add<0x114, 0xf>(v); v = dpp_add<0x118, 0xf>(v);
    v = dpp_add<0x142, 0xa>(v);
    return dpp_add<0x143, 0xc>(v);
}
DI unsigned bfi(unsigned m, unsigned a, unsigned b) { return (a & m) | (b & ~m); }

DI void encode_unit(const u32x4 (&x)[4], LAS uint8_t* slot, LAS unsigned* metaw, LAS uint8_t* scr, int lane) {
    *(LAS unsigned*)(slot + 3072 + lane * 4) = 0u;
    u32x4 t[4];
#pragma unroll
    for (int j = 0; j < 4; ++j) t[j] = x[j] & 0x7f7f7f7fu;
    unsigned sum = 0;
#pragma unroll
    for (int j = 0; j < 4; ++j) sum = __builtin_amdgcn_sad_u8(t[j][j], 0u, sum);
    const int bi = ((int)__builtin_amdgcn_readlane(wave_scan(sum), 63) - 20685) >> 10;
    const unsigned base = (unsigned)(bi < 0 ? 0 : bi > 96 ? 96 : bi);
    const unsigned k4 = (128u - base) * 0x01010101u;
    unsigned g[4];
#pragma unroll
    for (int j = 0; j < 4; ++j) {
        t[j] += k4;
        *(LAS u32x4*)(scr + j * 1024 + lane * 16) = t[j];
        const u32x4 fl = (~t[j] | (t[j] - 0x20202020u)) & 0x80808080u;
        const unsigned lo = __builtin_amdgcn_udot4(fl[1], 0x80402010u, __builtin_amdgcn_udot4(fl[0], 0x08040201u, 0u, false), false);
        const unsigned hi = __builtin_amdgcn_udot4(fl[3], 0x80402010u, __builtin_amdgcn_udot4(fl[2], 0x08040201u, 0u, false), false);
        g[j] = (hi << 8) + lo;
    }
    unsigned m0 = (g[0] >> 7) | (g[1] << 9), m1 = (g[2] >> 7) | (g[3] << 9);
    const unsigned c0 = __builtin_popcount(m0), c = c0 + __builtin_popcount(m1);
    const unsigned incl = wave_scan(c), cnt = __builtin_amdgcn_readlane(incl, 63);
    unsigned meta;
    if (cnt <= MAXEXC) {
        LAS uint8_t* a0 = slot + 3072 + (incl - c) * 2; LAS uint8_t* a1 = a0 + c0 * 2;
        LAS uint8_t* dump = scr + 4096;
        const unsigned l16 = lane * 16;
        auto round = [&]() {
            const unsigned b0 = __builtin_ctz(m0 | 0x80000000u), b1 = __builtin_ctz(m1 | 0x80000000u);
            *(LAS unsigned short*)(m0 ? a0 : dump) = (unsigned short)(l16 + (b0 & 15u) + ((b0 & 16u) << 6));
            *(LAS unsigned short*)(m1 ? a1 : dump) = (unsigned short)(l16 + 2048u + (b1 & 15u) + ((b1 & 16u) << 6));
            m0 &= m0 - 1u; m1 &= m1 - 1u; a0 += 2; a1 += 2;
        };
        round(); round(); round();
        while (__builtin_amdgcn_ballot_w64((m0 | m1) != 0u)) round();
        const unsigned e = *(LAS unsigned*)(slot + 3072 + lane * 4);
        const unsigned tl = scr[e & 0xfffu], th = scr[e >> 16];
        u32x4 a, b, cc;
#pragma unroll
        for (int d = 0; d < 4; ++d) {
            const unsigned td = t[3][d];
            a[d] = bfi(0x60606060u, td << 5, bfi(0x1f1f1f1fu, t[0][d], x[0][d]));
            b[d] = bfi(0x60606060u, td << 3, bfi(0x1f1f1f1fu, t[1][d], x[1][d]));
            cc[d] = bfi(0x60606060u, bfi(0x20202020u, td << 1, x[3][d] >> 1), bfi(0x1f1f1f1fu, t[2][d], x[2][d]));
        }
        *(LAS u32x4*)(slot + l16) = a; *(LAS u32x4*)(slot + 1024 + l16) = b; *(LAS u32x4*)(slot + 2048 + l16) = cc;
        const unsigned ql = (unsigned)lane * 2u < cnt ? ((tl ^ 0x80u) >> 5) << 13 : 0u, qh = (unsigned)lane * 2u + 1u < cnt ? ((th ^ 0x80u) >> 5) << 29 : 0u;
        *(LAS unsigned*)(slot + 3072 + lane * 4) = e | ql | qh;
        meta = base | (cnt << 8);
    } else {
#pragma unroll
        for (int j = 0; j < 4; ++j) *(LAS u32x4*)(slot + j * 1024 + lane * 16) = x[j];
        meta = META_RAW;
    }
    asm volatile("s_waitcnt lgkmcnt(0)" ::: "memory");
    if (lane == 0) *(volatile LAS unsigned*)metaw = meta | META_READY;
}

#define PACE_NUM 2643
#define NRAW 1
struct Unit4 { u32x4 g[4]; };
DI void unit_gather(Unit4& x, const uint8_t* s, unsigned o0, unsigned o1) {
    asm volatile("global_load_dwordx4 %0, %4, %6\n\tglobal_load_dwordx4 %1, %5, %6\n\tglobal_load_dwordx4 %2, %4, %7\n\tglobal_load_dwordx4 %3, %5, %7"
                 : "=&v"(x.g[0]), "=&v"(x.g[1]), "=&v"(x.g[2]), "=&v"(x.g[3]) : "v"(o0), "v"(o1), "s"(s), "s"(s + 16 * KDIM) : "memory");
}
DI void unit_gather_if(Unit4& x, bool on, const uint8_t* s, unsigned o0, unsigned o1) {
    asm volatile("s_cmp_eq_u32 %8, 0\n\ts_cbranch_scc1 1f\n\t"
                 "global_load_dwordx4 %0, %4, %6\n\tglobal_load_dwordx4 %1, %5, %6\n\tglobal_load_dwordx4 %2, %4, %7\n\tglobal_load_dwordx4 %3, %5, %7\n1:"
                 : "+v"(x.g[0]), "+v"(x.g[1]), "+v"(x.g[2]), "+v"(x.g[3]) : "v"(o0), "v"(o1), "s"(s), "s"(s + 16 * KDIM), "s"(rfl(on ? 1u : 0u)) : "memory", "scc");
}
DI void unit_send(const Unit4& x, uint8_t* d, unsigned voff, unsigned long long mask) {
    unsigned long long keep;
    asm volatile("global_store_dwordx4 %1, %2, %6 sc0 sc1\n\t"
                 "global_store_dwordx4 %1, %3, %6 offset:1024 sc0 sc1\n\t"
                 "s_mov_b64 %0, exec\n\t"
                 "global_store_dwordx4 %1, %4, %6 offset:2048 sc0 sc1\n\t"
                 "s_mov_b64 exec, %7\n\t"
                 "global_store_dwordx4 %1, %5, %6 offset:3072 sc0 sc1\n\t"
                 "s_mov_b64 exec, %0\n\t"
                 "s_nop 1"
                 : "=&s"(keep) : "v"(voff), "v"(x.g[0]), "v"(x.g[1]), "v"(x.g[2]), "v"(x.g[3]), "s"(d), "s"(mask) : "memory");
}
DI void comm_role(const Params& p, int wave, int lane, LAS uint8_t* lds, Stamps& st) {
    const int b = blockIdx.x, n = (512 - b + NCOMM - 1) / NCOMM;
    const unsigned q = b & 3u;
    const unsigned o0 = (lane >> 3) * KDIM + (((lane & 7) ^ (lane >> 4)) << 4), o1 = (8 + (lane >> 3)) * KDIM + (((lane & 7) ^ (4 + (lane >> 4))) << 4);
    auto chunk = [&](int i) { const unsigned o = (b >> 2) + i * (NCOMM / 4); return (p.order[o >> 2] >> ((o & 3u) * 8u)) & 0xffu; };
    const uint8_t* px = p.x;
    auto source = [&](unsigned c) { return px + (size_t)((c & 3u) * 128 + q * 32) * KDIM + (c >> 2) * 128; };
    Unit4 x0 = {}, xe = {};
    const bool sender = wave != 7;
    unsigned c0 = chunk(0);
    uint8_t* mw = p.myws; int rank = p.rank;
    uint8_t* pw = p.peer[wave];
    asm volatile("" : "+s"(c0), "+s"(mw), "+s"(rank), "+s"(px), "+s"(pw));
    unit_gather_if(x0, sender, source(c0), o0, o1);
    unsigned craw; call_issue(craw, mw, b * 8 + wave, sender ? lane : 1);
    unsigned c1 = chunk(NRAW + wave);
    LAS unsigned* metaw = (LAS unsigned*)(lds + CL_META);
    LAS uint8_t* scr = lds + CL_SCR + wave * 4160;
    if (wave == 7 && lane < 32) metaw[lane] = 0u;
    asm volatile("s_waitcnt lgkmcnt(0)\n\ts_barrier" : "+s"(c1), "+s"(pw) :: "memory");
    Unit4 xn;
    unit_gather(xn, source(c1), o0, o1);
    unsigned call = 0, due = (unsigned)(b * (4160 / NCOMM) * PACE_NUM);
    uint8_t* dbase = nullptr; uint8_t* fbase = nullptr;
    unsigned long long t0 = 0;
    auto now = [&]() { return (unsigned)(__builtin_amdgcn_s_memrealtime() - t0); };
    auto send = [&](unsigned c, unsigned meta) {
        const unsigned nl = (meta & META_RAW) ? 64u : (((meta >> 8) & 0xffu) + 7u) >> 3;
        while (now() < (due >> 16)) __builtin_amdgcn_s_sleep(1);
        unit_send(x0, dbase + (size_t)(((c & 3u) * 32 + (c >> 2)) * 16384), lane * 16, ~0ull >> (64u - max(nl, 1u)));
        due += (3072u + 64u + ((nl + 3u) >> 2) * 64u) * PACE_NUM;
    };
    auto flag = [&](unsigned c, unsigned meta) { if (lane == 0) st8_sys(fbase + ((c & 3u) * 128 + (c >> 2)) * 8, u32x2{call, meta}); };
    asm volatile("s_waitcnt vmcnt(4)" : "+v"(x0.g[0]), "+v"(x0.g[1]), "+v"(x0.g[2]), "+v"(x0.g[3]), "+v"(craw) :: "memory");
    if (sender) {
        call = call_get(craw);
        dbase = pw + WS_DATA + ((size_t)(call & 1u) * NRANK + rank) * XBYTES + q * 4096;
        fbase = pw + WS_FLAG + (size_t)((call & 1u) * NRANK + rank) * 4096 + q * 256;
        t0 = __builtin_amdgcn_s_memrealtime();
        STAMP(1);
        send(c0, META_RAW);
        STAMP(4);
    }
    unsigned cp = c0, mp = META_RAW;
    for (int i = sender ? 1 : 0; i <= n; ++i) {
        const int ie = 8 * (i - 1) + NRAW + wave;
        if (ie < n) {
            asm volatile("s_cmp_eq_u32 %4, 0\n\ts_cbranch_scc1 2f\n\ts_cmp_lg_u32 %5, 7\n\ts_cbranch_scc1 1f\n\ts_waitcnt vmcnt(0)\n1:\n\t"
                         "s_cmp_lg_u32 %4, 1\n\ts_cbranch_scc1 2f\n\ts_waitcnt vmcnt(4)\n2:"
                         : "+v"(xn.g[0]), "+v"(xn.g[1]), "+v"(xn.g[2]), "+v"(xn.g[3]) : "s"(i), "s"(wave) : "memory", "scc");
            if (i > 0) xe = xn;
            asm volatile("" : "+v"(xe.g[0]), "+v"(xe.g[1]), "+v"(xe.g[2]), "+v"(xe.g[3]));
            unit_gather_if(xn, i > 0 && ie + 8 < n, source(chunk(min(ie + 8, n - 1))), o0, o1);
            const int slot = i > 0 ? ie : 23;
            encode_unit(xe.g, lds + slot * 4096, metaw + slot, scr, lane);
            if (i == 1) STAMP(3);
        } else if (!sender) break;
        if (!sender) continue;
        unsigned meta = META_RAW, c = 0;
        if (i < n) c = chunk(i);
        if (i < NRAW) {
            Unit4 xr;
            unit_gather(xr, source(c), o0, o1);
            asm volatile("s_waitcnt vmcnt(0)" : "+v"(xr.g[0]), "+v"(xr.g[1]), "+v"(xr.g[2]), "+v"(xr.g[3]) :: "memory");
            x0 = xr;
            flag(cp, mp);
            send(c, meta);
        } else {
            if (i < n) meta = rfl(*(volatile LAS unsigned*)(metaw + i));
            auto fetch = [&]() {
                asm volatile("" ::: "memory");
#pragma unroll
                for (int j = 0; j < 4; ++j) x0.g[j] = *(LAS u32x4*)(lds + i * 4096 + j * 1024 + lane * 16);
            };
            if (meta & META_READY) fetch();
            waitv<0>();
            flag(cp, mp);
            if (i < n) {
                if (!(meta & META_READY)) {
                    do {
                        __builtin_amdgcn_s_sleep(1);
                        meta = rfl(*(volatile LAS unsigned*)(metaw + i));
                    } while (!(meta & META_READY));
                    fetch();
                }
                send(c, meta);
            }
        }
        if (i == 1) STAMP(5);
        if (i == n - 1) STAMP(6);
        cp = c; mp = meta;
    }
}

struct Tile { int rb, cb; };
struct XRegs { u32x4 r[4]; unsigned e; };
DI void unit_load(XRegs& R, unsigned meta, const unsigned (&vx)[4], unsigned ve, __amdgpu_buffer_rsrc_t srd, unsigned so) {
    asm volatile("buffer_load_dwordx4 %0, %5, %10, %11 offen sc0 sc1\n\t"
                 "buffer_load_dwordx4 %1, %6, %10, %11 offen sc0 sc1\n\t"
                 "s_bitcmp1_b32 %12, 16\n\t"
                 "buffer_load_dwordx4 %2, %7, %10, %11 offen sc0 sc1\n\t"
                 "s_cbranch_scc0 1f\n\t"
                 "buffer_load_dwordx4 %3, %8, %10, %11 offen sc0 sc1\n\t"
                 "s_branch 2f\n"
                 "1:\n\t"
                 "buffer_load_sshort %4, %9, %10, %11 offen sc0 sc1\n"
                 "2:"
                 : "+v"(R.r[0]), "+v"(R.r[1]), "+v"(R.r[2]), "+v"(R.r[3]), "+v"(R.e)
                 : "v"(vx[0]), "v"(vx[1]), "v"(vx[2]), "v"(vx[3]), "v"(ve), "s"(srd), "s"(so), "s"(meta) : "memory", "scc");
}
DI void loader_role(const Params& p, const Tile tl, int li, int lane, LAS uint8_t* lds, Stamps& st) {
#define LOADER_PRIO 3
    __builtin_amdgcn_s_setprio(LOADER_PRIO);
    unsigned craw; call_issue(craw, p.myws, blockIdx.x * 8 + 4 + li, lane);
    const bool remote = tl.rb < 28;
    const int src = remote ? ((p.rank + 1 + (tl.rb % 7)) & 7) : p.rank, sub = remote ? tl.rb / 7 : 0;
    uint8_t* myws = p.myws;
    const __amdgpu_buffer_rsrc_t sE = mk_srd(p.x + (size_t)(tl.rb < NHEAVY ? 160 + 32 * tl.rb : 128) * KDIM, 32 * KDIM);
    const __amdgpu_buffer_rsrc_t sW = mk_srd(p.w + tl.cb * 256, (unsigned)(KDIM - 1) * NFULL + 256);
    const unsigned rowm0 = (lane >> 3) * KDIM + (((lane & 7) ^ (lane >> 4)) << 4), rowm1 = (lane >> 3) * KDIM + (((lane & 7) ^ (4 + (lane >> 4))) << 4);
    const unsigned ve = li * 8 * KDIM + ((li & 1) ? rowm1 : rowm0);
    unsigned vx[4], vw[8];
#pragma unroll
    for (int j = 0; j < 4; ++j) vx[j] = remote ? li * 4096 + j * 1024 + lane * 16 : (li * 4 + j) * 8 * KDIM + ((j & 1) ? rowm1 : rowm0);
    const unsigned vxe = li * 4096 + 3072 + lane * 2;
    const unsigned xk = remote ? 16384 : 128;
#pragma unroll
    for (int i = 0; i < 8; ++i) {
        const unsigned k = 32 * li + 4 * i + (lane >> 4);
        vw[i] = k * NFULL + (((lane & 15) ^ ((k & 7) << 1)) << 4) - (i & 3) * 1024;
    }
    const unsigned ldsb = (unsigned)(uintptr_t)lds;
    const unsigned mE = ldsb + 16384 + li * 1024, mW = ldsb + XS + li * 8192;
#define WPIECE(i, t, so) do { if constexpr ((i) == 0) { set_m0(mE + (so)); dma<0>(ve, sE, (t) * 128); } \
        else { if constexpr ((i) == 1 || (i) == 5) set_m0(mW + (so) + ((i) - 1) * 1024); dma<(((i) - 1) & 3) * 1024>(vw[(i) - 1], sW, (t) * (128 * NFULL)); } } while (0)
    auto issueW = [&](int t, unsigned so) {
        WPIECE(0, t, so); WPIECE(1, t, so); WPIECE(2, t, so); WPIECE(3, t, so); WPIECE(4, t, so); WPIECE(5, t, so); WPIECE(6, t, so); WPIECE(7, t, so); WPIECE(8, t, so);
    };
    asm volatile("s_nop 4" ::: "memory");
    issueW(0, 0);
    waitv<9>();
    const unsigned call = call_get(craw);
    const uint8_t* xb = remote ? myws + WS_DATA + (((size_t)(call & 1u) * NRANK + src) * 4 + sub) * (XBYTES / 4) : p.x;
    const __amdgpu_buffer_rsrc_t sX = mk_srd(xb, (unsigned)(XBYTES / 4));
    const uint8_t* fl = myws + WS_FLAG + (size_t)((call & 1u) * NRANK + src) * 4096 + ((sub * 4 + li) * 32 + (lane & 31)) * 8;
    asm volatile("s_nop 4" ::: "memory");

    int avail = remote ? 0 : 64; unsigned dead = 0;
    unsigned metav = META_RAW;
    u32x2 pv = {0u, 0u};
    XRegs R0 = {}, R1 = {};
    auto poll_async = [&](int lim) {
        asm volatile("s_cmp_gt_i32 %2, %3\n\ts_cbranch_scc1 1f\n\tglobal_load_dwordx2 %0, %1, off sc0 sc1\n1:" : "+v"(pv) : "v"(fl), "s"(avail), "s"(lim) : "memory", "scc");
    };
    auto take = [&](const u32x2& v) {
        const unsigned long long m = __builtin_amdgcn_ballot_w64(v.x < call);
        avail = m ? __builtin_ctzll(m) : 64;
        metav = v.y;
    };
    auto need = [&](int s) {
        asm volatile("s_cmp_gt_i32 %1, %2\n\ts_cbranch_scc1 1f\n\ts_waitcnt vmcnt(0)\n1:" : "+v"(pv) : "s"(avail), "s"(s) : "memory", "scc");
        if (avail > s) return;
        take(pv);
        if (avail <= s) {
            const unsigned long long t0 = __builtin_amdgcn_s_memrealtime();
            do {
                u32x2 v;
                asm volatile("global_load_dwordx2 %0, %1, off sc0 sc1\n\ts_waitcnt vmcnt(0)" : "=&v"(v) : "v"(fl) : "memory");
                take(v);
                if (__builtin_amdgcn_s_memrealtime() - t0 > DEADLINE_TICKS && avail <= s) { dead = 1; avail = 64; metav = META_RAW; }
            } while (avail <= s);
        }
    };
    unsigned m60 = 0x60606060u; asm("" : "+s"(m60));
    auto decode = [&](auto withw, int s, const XRegs& R, unsigned meta, unsigned so, unsigned sn) {
        constexpr bool WW = decltype(withw)::value;
        LAS uint8_t* img = lds + so + li * 4096;
        if (__builtin_expect(meta & META_RAW, 0)) {
            if constexpr (WW) { WPIECE(0, s + 1, sn); WPIECE(1, s + 1, sn); }
            *(LAS u32x4*)(img + lane * 16) = R.r[0];
            if constexpr (WW) { WPIECE(2, s + 1, sn); WPIECE(3, s + 1, sn); }
            *(LAS u32x4*)(img + 1024 + lane * 16) = R.r[1];
            if constexpr (WW) { WPIECE(4, s + 1, sn); WPIECE(5, s + 1, sn); }
            *(LAS u32x4*)(img + 2048 + lane * 16) = R.r[2];
            if constexpr (WW) { WPIECE(6, s + 1, sn); WPIECE(7, s + 1, sn); }
            *(LAS u32x4*)(img + 3072 + lane * 16) = R.r[3];
            if constexpr (WW) WPIECE(8, s + 1, sn);
        } else {
            const unsigned base4 = (meta & 0xffu) * 0x01010101u, tab = base4 + 0x90801000u;
            const unsigned nx = (meta >> 8) & 0xffu;
            auto patch = [&](unsigned e) {
                __hip_atomic_fetch_add((LAS unsigned*)(img + (e & 0xffcu)), (unsigned)((int)e >> 13) << (((e & 3u) << 3) + 5u), __ATOMIC_RELAXED, __HIP_MEMORY_SCOPE_WORKGROUP);
            };
            u32x4 sA, sB, sC;
            sA = R.r[0] & m60; asm("" : "+v"(sA));
            *(LAS u32x4*)(img + lane * 16) = (R.r[0] ^ sA) + base4;
            if constexpr (WW) { GAP(); WPIECE(0, s + 1, sn); GAP(); }
            sB = R.r[1] & m60; asm("" : "+v"(sB));
            *(LAS u32x4*)(img + 1024 + lane * 16) = (R.r[1] ^ sB) + base4;
            if constexpr (WW) { GAP(); WPIECE(1, s + 1, sn); GAP(); }
            sC = R.r[2] & m60; asm("" : "+v"(sC));
            *(LAS u32x4*)(img + 2048 + lane * 16) = (R.r[2] ^ sC) + base4;
            if constexpr (WW) { GAP(); WPIECE(2, s + 1, sn); GAP(); }
            u32x4 o3;
            typedef unsigned long long u64;
#pragma unroll
            for (int d = 0; d < 4; d += 2) {
                u64 a = (((u64)sA[d + 1] << 32) | sA[d]) >> 5, b2 = (((u64)sB[d + 1] << 32) | sB[d]) >> 3, c = (((u64)sC[d + 1] << 32) | sC[d]) >> 5;
                asm("" : "+v"(a), "+v"(b2), "+v"(c));
                o3[d] = __builtin_amdgcn_perm(tab, tab, (unsigned)c) + (unsigned)a + (unsigned)b2;
                o3[d + 1] = __builtin_amdgcn_perm(tab, tab, (unsigned)(c >> 32)) + (unsigned)(a >> 32) + (unsigned)(b2 >> 32);
                if constexpr (WW) { if (d == 0) { GAP(); WPIECE(3, s + 1, sn); GAP(); } }
            }
            *(LAS u32x4*)(img + 3072 + lane * 16) = o3;
            if constexpr (WW) { GAP(); WPIECE(4, s + 1, sn); GAP(); }
            patch((unsigned)lane < nx ? R.e : (unsigned)lane * 4u);
            if (__builtin_expect(nx > 64u, 0)) {
                unsigned e2;
                asm volatile("buffer_load_sshort %0, %1, %2, %3 offen offset:128 sc0 sc1\n\ts_waitcnt vmcnt(0)" : "=&v"(e2) : "v"(vxe), "s"(sX), "s"(s * xk) : "memory");
                patch((unsigned)lane + 64u < nx ? e2 : (unsigned)lane * 4u);
            }
            if constexpr (WW) { GAP(); WPIECE(5, s + 1, sn); WPIECE(6, s + 1, sn); WPIECE(7, s + 1, sn); WPIECE(8, s + 1, sn); }
        }
    };
    unsigned so = 0, mc, mn;
    poll_async(0);
    need(0);
    mc = __builtin_amdgcn_readlane(metav, 0);
    unit_load(R0, mc, vx, vxe, sX, 0);
    poll_async(1);
    waitv<0>();
    auto step = [&](int s, XRegs& Rc, XRegs& Rn) {
        const unsigned sn = so == 2 * STG ? 0 : so + STG;
        need(s + 1);
        SEG(1);
        mn = __builtin_amdgcn_readlane(metav, s + 1);
        unit_load(Rn, mn, vx, vxe, sX, (s + 1) * xk);
        SEG(2);
        asm volatile("s_waitcnt vmcnt(13)" : "+v"(Rc.r[0]), "+v"(Rc.r[1]), "+v"(Rc.r[2]), "+v"(Rc.r[3]), "+v"(Rc.e) :: "memory");
        SEG(3);
        decode(std::true_type{}, s, Rc, mc, so, sn);
        SEG(4);
        waitv<13>();
        poll_async(s + 2);
        so = sn; mc = mn;
        asm volatile("s_waitcnt lgkmcnt(0)" ::: "memory");
        SEG(5);
        barrier();
        SEG(6);
    };
    for (int s = 0; s < KSTEPS - 2; s += 2) { step(s, R0, R1); step(s + 1, R1, R0); }
    step(KSTEPS - 2, R0, R1);
    waitv<9>();
    asm volatile("" : "+v"(R1.r[0]), "+v"(R1.r[1]), "+v"(R1.r[2]), "+v"(R1.r[3]), "+v"(R1.e));
    decode(std::false_type{}, KSTEPS - 1, R1, mc, so, 0);
    if (lane == 0) *(LAS unsigned*)(lds + LDS_DEAD + li * 4) = dead;
    asm volatile("s_waitcnt vmcnt(0) lgkmcnt(0)" : "+v"(pv) :: "memory");
    barrier();
    STAMP(5);
    STAMP_FLUSH();
}

DI v8i rd_x(LAS uint8_t* a0, LAS uint8_t* a1, int mt) {
    const v4i lo = *(LAS v4i*)(a0 + mt * 4096), hi = *(LAS v4i*)(a1 + mt * 4096);
    return __builtin_shufflevector(lo, hi, 0, 1, 2, 3, 4, 5, 6, 7);
}
DI v8i rd_w(LAS uint8_t* a, int kh) {
    const v2i r0 = __builtin_amdgcn_ds_read_tr8_b64_v2i32((LAS v2i*)(a + XS + kh * 16384));
    const v2i r1 = __builtin_amdgcn_ds_read_tr8_b64_v2i32((LAS v2i*)(a + XS + kh * 16384 + 2048));
    const v2i r2 = __builtin_amdgcn_ds_read_tr8_b64_v2i32((LAS v2i*)(a + XS + kh * 16384 + 4096));
    const v2i r3 = __builtin_amdgcn_ds_read_tr8_b64_v2i32((LAS v2i*)(a + XS + kh * 16384 + 6144));
    const v4i lo = __builtin_shufflevector(r0, r1, 0, 1, 2, 3), hi = __builtin_shufflevector(r2, r3, 0, 1, 2, 3);
    return __builtin_shufflevector(lo, hi, 0, 1, 2, 3, 4, 5, 6, 7);
}
#define MM(mt, nt, kh, xf) do { acc[mt][nt] = __builtin_amdgcn_mfma_scale_f32_32x32x64_f8f6f4(W[nt][kh], xf, acc[mt][nt], 1, 1, 0, 0, 0, 0); GAP(); } while (0)

template <int NM>
DI void mfma_role(const Params& p, const Tile tl, int w, int lane, LAS uint8_t* lds, Stamps& st) {
    const int r = lane & 31, g = lane >> 5, q = (lane & 15) >> 1;
    f32x16 acc[NM][2];
#pragma unroll
    for (int m = 0; m < NM; ++m) { acc[m][0] = (f32x16){}; acc[m][1] = (f32x16){}; }
    v8i W[2][2], xp0 = {}, xp1 = {};
    W[0][1] = (v8i){}; W[1][1] = (v8i){};
    const int xa = r * 128 + (((2 * g) ^ ((r >> 1) & 7)) << 4);
    const int wa = (32 * g + q) * 256 + (((4 * w + ((lane >> 4) & 1)) ^ (q << 1)) << 4) + 8 * (lane & 1);
    float sc = p.sx[0] * p.sw[0];
    asm volatile("" : "+v"(sc));
    unsigned row_main = tl.rb < 28 ? ((p.rank + 1 + (tl.rb % 7)) & 7) * MLOC + (tl.rb / 7) * 128 : p.rank * MLOC;
    unsigned row_extra = p.rank * MLOC + (tl.rb < NHEAVY ? 160 + 32 * tl.rb : 128);
    __amdgpu_buffer_rsrc_t sO = mk_srd(p.out, 4096u * NLOC * 4u);
    asm volatile("" : "+s"(row_main), "+s"(row_extra), "+s"(sO));
    int so = 0;
    for (int t = 0; t < KSTEPS; ++t) {
        LAS uint8_t* a00 = lds + (so + xa); LAS uint8_t* a01 = lds + (so + (xa ^ 16)); LAS uint8_t* a10 = lds + (so + (xa ^ 64)); LAS uint8_t* a11 = lds + (so + (xa ^ 80));
        LAS uint8_t* w0 = lds + (so + wa); LAS uint8_t* w1 = lds + (so + (wa ^ 32));
        asm volatile("" : "+v"(a00), "+v"(a01), "+v"(a10), "+v"(a11), "+v"(w0), "+v"(w1));
        so = (so == 2 * STG) ? 0 : so + STG;
        barrier(); GAP();
        v8i xf[2 * NM];
        MM(NM - 2, 0, 1, xp0);
        xf[0] = rd_x(a00, a01, 0); GAP();
        W[0][0] = rd_w(w0, 0); GAP();
        MM(NM - 2, 1, 1, xp0);
        W[1][0] = rd_w(w1, 0); GAP();
        MM(NM - 1, 0, 1, xp1);
        xf[1] = rd_x(a00, a01, 1); GAP();
        MM(NM - 1, 1, 1, xp1);
#pragma unroll
        for (int f = 0; f < 2 * NM - 2; ++f) {
            const int fn = f + 2;
            xf[fn] = fn < NM ? rd_x(a00, a01, fn) : rd_x(a10, a11, fn - NM); GAP();
            MM(f % NM, 0, f / NM, xf[f]);
            if (f == 0) { W[0][1] = rd_w(w0, 1); GAP(); }
            if (f == 1) { W[1][1] = rd_w(w1, 1); GAP(); }
            MM(f % NM, 1, f / NM, xf[f]);
        }
        xp0 = xf[2 * NM - 2]; xp1 = xf[2 * NM - 1];
        asm volatile("s_waitcnt lgkmcnt(0)" : "+v"(xp0), "+v"(xp1) :: "memory"); GAP();
    }
    const u32x4 dd = *(LAS u32x4*)(lds + LDS_DEAD);
    MM(NM - 2, 0, 1, xp0); MM(NM - 2, 1, 1, xp0); MM(NM - 1, 0, 1, xp1); MM(NM - 1, 1, 1, xp1);
    STAMP(1);
    if (dd.x | dd.y | dd.z | dd.w) sc = __builtin_nanf("");
    LAS uint8_t* eb = lds + 2 * STG + w * 12288;
    const int ew = r * 128 + ((g ^ (r & 7)) << 4), er = (lane >> 3) * 128 + (((lane & 7) ^ (lane >> 3)) << 4);
    const unsigned vo = ((lane >> 3) * NLOC + tl.cb * 256 + w * 64 + 4 * (lane & 7)) * 4;
    auto stage = [&](int i, u32x4 (&v)[4]) {
        LAS uint8_t* e = eb + (i % 3) * 4096;
        const f32x16 a = acc[i >> 1][i & 1] * sc;
#pragma unroll
        for (int k = 0; k < 4; ++k) *(LAS f32x4*)(e + (ew ^ (k << 5))) = (f32x4){a[4 * k], a[4 * k + 1], a[4 * k + 2], a[4 * k + 3]};
#pragma unroll
        for (int k = 0; k < 4; ++k) v[k] = *(LAS u32x4*)(e + er + k * 1024);
    };
    auto put = [&](int i, u32x4 (&v)[4]) {
        const unsigned row = (i >> 1) < 4 ? row_main + 32 * (i >> 1) : row_extra;
#pragma unroll
        for (int k = 0; k < 4; ++k) __builtin_amdgcn_raw_buffer_store_b128(v[k], sO, vo, (row + 8 * k) * (NLOC * 4) + (i & 1) * 128, OUT_AUX);
    };
    u32x4 ev[2][4];
    stage(0, ev[0]);
#pragma unroll
    for (int i = 1; i < 2 * NM; ++i) { stage(i, ev[i & 1]); GAP(); put(i - 1, ev[(i - 1) & 1]); GAP(); }
    put(2 * NM - 1, ev[1]);
    STAMP(2);
    STAMP(3); STAMP_FLUSH();
}

__global__ __launch_bounds__(512, 2) void gemm_ag_kernel(const Params p) {
    extern __shared__ __attribute__((aligned(16))) uint8_t smem[];
    LAS uint8_t* lds = (LAS uint8_t*)smem;
    const int lane = threadIdx.x & 63, wave = rfl(threadIdx.x >> 6), b = blockIdx.x;
    Stamps st = {};
    STAMP(0);
    if (__builtin_expect(b < NCOMM, 1)) {
        comm_role(p, wave, lane, lds, st);
        return;
    }
    Tile tl;
    { const int xcd = b & 7, j = (b - NCOMM) >> 3;
      tl.rb = j; tl.cb = xcd;
    }
    if (wave == 0) {
        const unsigned j = (unsigned)(b - NCOMM) >> 3, k = j / 7u;
        if (k < 3u) {
            const uint8_t* t = p.peer[j - 7u * k] + (k == 2u ? WS_FLAG : WS_DATA + (size_t)(k * NRANK + p.rank) * XBYTES);
            unsigned d;
            asm volatile("global_load_dword %0, %1, %2 sc0 sc1\n\ts_waitcnt vmcnt(0)" : "=&v"(d) : "v"(0u), "s"(t) : "memory");
        }
    }
    if (wave >= 4) {
        loader_role(p, tl, wave - 4, lane, lds, st);
    } else {
        if (tl.rb < NHEAVY || tl.rb == 28) mfma_role<5>(p, tl, wave, lane, lds, st);
        else mfma_role<4>(p, tl, wave, lane, lds, st);
    }
}

}

extern "C" void kernel_launch(int rank, int nranks, ncclComm_t comm, hipStream_t stream,
                              void* const* d_in, const int* in_sizes, int n_in,
                              void* d_out, int out_size,
                              const int* in_dtypes, int out_dtype,
                              void* const* d_out_all,
                              void* const* d_ws_all, size_t ws_bytes) {
    if (nranks != NRANK || ws_bytes < WS_NEED || (unsigned)rank >= (unsigned)NRANK) { fprintf(stderr, "unexpected configuration\n"); return; }
    static bool attr_set = false;
    if (!attr_set) {
        HIP_CHECK(hipFuncSetAttribute((const void*)gemm_ag_kernel, hipFuncAttributeMaxDynamicSharedMemorySize, LDS_BYTES));
        attr_set = true;
    }
    Params p;
    p.x = (const uint8_t*)d_in[0]; p.w = (const uint8_t*)d_in[1] + (size_t)rank * NLOC; p.sx = (const float*)d_in[2]; p.sw = (const float*)d_in[3];
    p.out = (float*)d_out;
    p.myws = (uint8_t*)d_ws_all[rank];
    for (int j = 0; j < NRANK; ++j) p.peer[j] = (uint8_t*)d_ws_all[(rank + 1 + j) % NRANK];
    p.rank = rank; p.pad = 0;
    {
#define STAGV 0, 1, 2, 3
        const int stag[4] = {STAGV};
        uint8_t ord[128]; int n = 0;
        for (int key = 0; key < 2 * KSTEPS; ++key)
            for (int sub = 0; sub < 4; ++sub) { const int ks = key - stag[sub]; if (ks >= 0 && ks < KSTEPS) ord[n++] = (uint8_t)(sub | (ks << 2)); }
        for (int i = 0; i < 32; ++i) p.order[i] = ord[4 * i] | (ord[4 * i + 1] << 8) | (ord[4 * i + 2] << 16) | ((unsigned)ord[4 * i + 3] << 24);
    }
    hipLaunchKernelGGL(gemm_ag_kernel, dim3(GRID), dim3(512), LDS_BYTES, stream, p);
    HIP_CHECK(hipGetLastError());
}
```

```cpp
#include <hip/hip_runtime.h>
#include <rccl/rccl.h>
#include <cstdio>
#include <cstdint>
#include <type_traits>

#define HIP_CHECK(expr) do { hipError_t _e = (expr); if (_e != hipSuccess) { \
    fprintf(stderr, "HIP error %s at %s:%d\n", hipGetErrorString(_e), __FILE__, __LINE__); \
    return; } } while (0)

namespace {

typedef int      v8i   __attribute__((ext_vector_type(8)));
typedef int      v4i   __attribute__((ext_vector_type(4)));
typedef int      v2i   __attribute__((ext_vector_type(2)));
typedef unsigned u32x4 __attribute__((ext_vector_type(4)));
typedef unsigned u32x2 __attribute__((ext_vector_type(2)));
typedef float    f32x4 __attribute__((ext_vector_type(4)));
typedef float    f32x16 __attribute__((ext_vector_type(16)));
#define LAS __attribute__((address_space(3)))
#define DI __device__ __forceinline__

constexpr int NRANK = 8;
constexpr int KDIM = 4096, NFULL = 16384, NLOC = 2048, MLOC = 512;
constexpr int KSTEPS = 32;
constexpr int XS = 160 * 128;
constexpr int NHEAVY = 11;
constexpr int STG = XS + 32768;
constexpr int LDS_DEAD = 3 * STG;
constexpr int LDS_BYTES = LDS_DEAD + 16;
constexpr int NCOMM = 24;
constexpr int GRID = 256;

constexpr size_t WS_CNT = 0;
constexpr size_t WS_FLAG = 128u << 10;
constexpr size_t WS_STAMP = 512u << 10;
constexpr size_t WS_TRACE = 1u << 20;
constexpr size_t WS_DATA = 2u << 20;
constexpr size_t XBYTES = (size_t)MLOC * KDIM;
constexpr size_t WS_NEED = WS_DATA + 2 * NRANK * XBYTES;
constexpr unsigned long long DEADLINE_TICKS = 50000000ull;

struct Params {
    const uint8_t* x;
    uint8_t* myws;
    const uint8_t* w;
    int rank;
    int pad;
    unsigned order[32];
    uint8_t* peer[NRANK];
    const float* sx;
    const float* sw;
    float* out;
};

DI unsigned rfl(unsigned v) { return __builtin_amdgcn_readfirstlane(v); }
template <int N> DI void waitv() { asm volatile("s_waitcnt vmcnt(%0)" :: "i"(N) : "memory"); }
DI void barrier() { asm volatile("s_barrier" ::: "memory"); }
#define OUT_AUX 16
DI void st8_sys(uint8_t* p, const u32x2& v) { asm volatile("global_store_dwordx2 %0, %1, off sc0 sc1\n\ts_nop 1" :: "v"(p), "v"(v) : "memory"); }
DI __amdgpu_buffer_rsrc_t mk_srd(const void* ptr, unsigned bytes) {
    const uintptr_t a = (uintptr_t)ptr;
    const uintptr_t u = ((uintptr_t)rfl((unsigned)(a >> 32)) << 32) | rfl((unsigned)a);
    return __builtin_amdgcn_make_buffer_rsrc((void*)u, 0, (int)rfl(bytes), 0x00020000);
}
DI void set_m0(unsigned m) { asm volatile("s_mov_b32 m0, %0\n\ts_nop 0" :: "s"(m) : "memory"); }
template <int OFF> DI void dma(unsigned voff, __amdgpu_buffer_rsrc_t srd, unsigned soff) {
    asm volatile("buffer_load_dwordx4 %0, %1, %2 offen offset:%3 lds" :: "v"(voff), "s"(srd), "s"(soff), "i"(OFF) : "memory");
}
template <int MI, int SI, int OFF> DI void dma2(unsigned mb, unsigned voff, __amdgpu_buffer_rsrc_t srd, unsigned sb) {
    unsigned tmp;
    asm volatile("s_add_u32 m0, %1, %5\n\ts_add_u32 %0, %4, %6\n\tbuffer_load_dwordx4 %2, %3, %0 offen offset:%7 lds"
                 : "=&s"(tmp) : "s"(mb), "v"(voff), "s"(srd), "s"(sb), "i"(MI), "i"(SI), "i"(OFF) : "memory", "scc");
}
#define GAP() __builtin_amdgcn_sched_barrier(0)
struct Stamps {};
#define STAMP(i) do {} while (0)
#define STAMP_FLUSH() do {} while (0)
#define SEG(i) do {} while (0)
constexpr int MAXEXC = 128;
constexpr unsigned META_RAW = 1u << 16, META_READY = 1u << 31;
constexpr int CL_META = 96 * 1024;
constexpr int CL_SCR = 100 * 1024;

template <int CTRL, int RM> DI unsigned dpp_add(unsigned v) { return v + __builtin_amdgcn_update_dpp(0u, v, CTRL, RM, 0xf, true); }
DI unsigned wave_scan(unsigned v) {
    v = dpp_add<0x111, 0xf>(v); v = dpp_add<0x112, 0xf>(v); v = dpp_add<0x114, 0xf>(v); v = dpp_add<0x118, 0xf>(v);
    v = dpp_add<0x142, 0xa>(v);
    return dpp_add<0x143, 0xc>(v);
}
DI unsigned bfi(unsigned m, unsigned a, unsigned b) { return (a & m) | (b & ~m); }

DI void encode_unit(LAS uint8_t* slot, LAS unsigned* metaw, LAS uint8_t* scr, int lane) {
    u32x4 x[4], t[4];
#pragma unroll
    for (int j = 0; j < 4; ++j) x[j] = *(LAS u32x4*)(slot + j * 1024 + lane * 16);
#pragma unroll
    for (int j = 0; j < 4; ++j) t[j] = x[j] & 0x7f7f7f7fu;
    unsigned sum = 0;
#pragma unroll
    for (int j = 0; j < 4; ++j) sum = __builtin_amdgcn_sad_u8(t[j][j], 0u, sum);
    const int bi = ((int)__builtin_amdgcn_readlane(wave_scan(sum), 63) - 20685) >> 10;
    const unsigned base = (unsigned)(bi < 0 ? 0 : bi > 96 ? 96 : bi);
    const unsigned k4 = (128u - base) * 0x01010101u;
    unsigned g[4];
#pragma unroll
    for (int j = 0; j < 4; ++j) {
        t[j] += k4;
        *(LAS u32x4*)(scr + j * 1024 + lane * 16) = t[j];
        const u32x4 fl = (~t[j] | (t[j] - 0x20202020u)) & 0x80808080u;
        const unsigned lo = __builtin_amdgcn_udot4(fl[1], 0x80402010u, __builtin_amdgcn_udot4(fl[0], 0x08040201u, 0u, false), false);
        const unsigned hi = __builtin_amdgcn_udot4(fl[3], 0x80402010u, __builtin_amdgcn_udot4(fl[2], 0x08040201u, 0u, false), false);
        g[j] = (hi << 8) + lo;
    }
    unsigned m0 = (g[0] >> 7) | (g[1] << 9), m1 = (g[2] >> 7) | (g[3] << 9);
    const unsigned c0 = __builtin_popcount(m0), c = c0 + __builtin_popcount(m1);
    const unsigned incl = wave_scan(c), cnt = __builtin_amdgcn_readlane(incl, 63);
    unsigned meta;
    if (cnt <= MAXEXC) {
        *(LAS unsigned*)(slot + 3072 + lane * 4) = 0u;
        LAS uint8_t* a0 = slot + 3072 + (incl - c) * 2; LAS uint8_t* a1 = a0 + c0 * 2;
        LAS uint8_t* dump = scr + 4096;
        const unsigned l16 = lane * 16;
        auto round = [&]() {
            const unsigned b0 = __builtin_ctz(m0 | 0x80000000u), b1 = __builtin_ctz(m1 | 0x80000000u);
            *(LAS unsigned short*)(m0 ? a0 : dump) = (unsigned short)(l16 + (b0 & 15u) + ((b0 & 16u) << 6));
            *(LAS unsigned short*)(m1 ? a1 : dump) = (unsigned short)(l16 + 2048u + (b1 & 15u) + ((b1 & 16u) << 6));
            m0 &= m0 - 1u; m1 &= m1 - 1u; a0 += 2; a1 += 2;
        };
        asm volatile("" ::: "memory");
        round(); round(); round();
        while (__builtin_amdgcn_ballot_w64((m0 | m1) != 0u)) round();
        asm volatile("" ::: "memory");
        const unsigned e = *(LAS unsigned*)(slot + 3072 + lane * 4);
        const unsigned tl = scr[e & 0xfffu], th = scr[e >> 16];
        u32x4 a, b, cc;
#pragma unroll
        for (int d = 0; d < 4; ++d) {
            const unsigned td = t[3][d];
            a[d] = bfi(0x60606060u, td << 5, bfi(0x1f1f1f1fu, t[0][d], x[0][d]));
            b[d] = bfi(0x60606060u, td << 3, bfi(0x1f1f1f1fu, t[1][d], x[1][d]));
            cc[d] = bfi(0x60606060u, bfi(0x20202020u, td << 1, x[3][d] >> 1), bfi(0x1f1f1f1fu, t[2][d], x[2][d]));
        }
        *(LAS u32x4*)(slot + l16) = a; *(LAS u32x4*)(slot + 1024 + l16) = b; *(LAS u32x4*)(slot + 2048 + l16) = cc;
        const unsigned ql = (unsigned)lane * 2u < cnt ? ((tl ^ 0x80u) >> 5) << 13 : 0u, qh = (unsigned)lane * 2u + 1u < cnt ? ((th ^ 0x80u) >> 5) << 29 : 0u;
        *(LAS unsigned*)(slot + 3072 + lane * 4) = e | ql | qh;
        meta = base | (cnt << 8);
    } else {
        meta = META_RAW;
    }
    asm volatile("s_waitcnt lgkmcnt(0)" ::: "memory");
    if (lane == 0) *(volatile LAS unsigned*)metaw = meta | META_READY;
}

#define PACE_NUM 2680
struct Unit4 { u32x4 g[4]; };
DI void unit_send(const Unit4& x, uint8_t* d, unsigned voff, unsigned long long mask) {
    unsigned long long keep;
    asm volatile("global_store_dwordx4 %1, %2, %6 sc0 sc1\n\t"
                 "global_store_dwordx4 %1, %3, %6 offset:1024 sc0 sc1\n\t"
                 "s_mov_b64 %0, exec\n\t"
                 "global_store_dwordx4 %1, %4, %6 offset:2048 sc0 sc1\n\t"
                 "s_mov_b64 exec, %7\n\t"
                 "global_store_dwordx4 %1, %5, %6 offset:3072 sc0 sc1\n\t"
                 "s_mov_b64 exec, %0\n\t"
                 "s_nop 1"
                 : "=&s"(keep) : "v"(voff), "v"(x.g[0]), "v"(x.g[1]), "v"(x.g[2]), "v"(x.g[3]), "s"(d), "s"(mask) : "memory");
}
DI void comm_role(const Params& p, int wave, int lane, LAS uint8_t* lds, Stamps& st) {
    const int b = blockIdx.x, n = (512 - b + NCOMM - 1) / NCOMM;
    const unsigned q = b & 3u;
    const unsigned o0 = (lane >> 3) * KDIM + (((lane & 7) ^ (lane >> 4)) << 4), o1 = (8 + (lane >> 3)) * KDIM + (((lane & 7) ^ (4 + (lane >> 4))) << 4);
    auto chunk = [&](int i) { const unsigned o = (b >> 2) + i * (NCOMM / 4); return (p.order[o >> 2] >> ((o & 3u) * 8u)) & 0xffu; };
    const uint8_t* px = p.x;
    auto source = [&](unsigned c) { return px + (size_t)((c & 3u) * 128 + q * 32) * KDIM + (c >> 2) * 128; };
    const bool sender = wave != 7;
    unsigned c0 = chunk(sender ? 0 : 1);
    uint8_t* mw = p.myws; int rank = p.rank;
    uint8_t* pw = p.peer[wave];
    asm volatile("" : "+s"(c0), "+s"(mw), "+s"(rank), "+s"(px), "+s"(pw));
    LAS unsigned* metaw = (LAS unsigned*)(lds + CL_META);
    if (!sender) {
        const __amdgpu_buffer_rsrc_t sS = mk_srd(px, (unsigned)XBYTES);
        const unsigned g1 = o1 - 1024, g2 = o0 + 16 * KDIM - 2048, g3 = o1 + 16 * KDIM - 3072;
        auto stage_in = [&](int i, unsigned c) {
            const unsigned so = ((c & 3u) * 128 + q * 32) * KDIM + (c >> 2) * 128;
            set_m0((unsigned)(uintptr_t)lds + i * 4096);
            dma<0>(o0, sS, so); dma<1024>(g1, sS, so); dma<2048>(g2, sS, so); dma<3072>(g3, sS, so);
        };
        asm volatile("s_nop 4" ::: "memory");
        stage_in(1, c0);
        metaw[lane] = 0u;
        asm volatile("s_waitcnt lgkmcnt(0)\n\ts_barrier" ::: "memory");
        stage_in(2, chunk(2));
        for (int i = 0; i < n; ++i) {
            stage_in(min(i + 3, 23), chunk(min(i + 3, n - 1)));
            waitv<12>();
            encode_unit(lds + i * 4096, metaw + i, lds + CL_SCR, lane);
            if (i == 1) STAMP(3);
        }
        waitv<0>();
        STAMP(4); STAMP_FLUSH();
        return;
    }
    Unit4 x0; unsigned craw;
    {
        const uint8_t* s0 = source(c0); unsigned long long keep;
        asm volatile("global_load_dwordx4 %0, %6, %8\n\tglobal_load_dwordx4 %1, %7, %8\n\tglobal_load_dwordx4 %2, %6, %9\n\tglobal_load_dwordx4 %3, %7, %9\n\t"
                     "s_mov_b64 %5, exec\n\ts_mov_b64 exec, 1\n\tglobal_atomic_add %4, %10, %11, %12 sc0\n\ts_mov_b64 exec, %5\n\t"
                     "s_barrier\n\ts_waitcnt vmcnt(0)"
                     : "=&v"(x0.g[0]), "=&v"(x0.g[1]), "=&v"(x0.g[2]), "=&v"(x0.g[3]), "=&v"(craw), "=&s"(keep)
                     : "v"(o0), "v"(o1), "s"(s0), "s"(s0 + 16 * KDIM), "v"((unsigned)(WS_CNT + (b * 8 + wave) * 64)), "v"(1u), "s"(mw) : "memory");
    }
    const unsigned call = rfl(craw) + 1u;
    uint8_t* const dbase = pw + WS_DATA + ((size_t)(call & 1u) * NRANK + rank) * XBYTES + q * 4096;
    uint8_t* const fbase = pw + WS_FLAG + (size_t)((call & 1u) * NRANK + rank) * 4096 + q * 256;
    const unsigned long long t0 = __builtin_amdgcn_s_memrealtime();
    STAMP(1);
    unsigned due = 0;
    unsigned c = c0, meta = META_RAW;
    for (int i = 0; i < n; ++i) {
        const unsigned nl = (meta & META_RAW) ? 64u : (((meta >> 8) & 0xffu) + 7u) >> 3;
        const unsigned cost = (48u + 1u + ((nl + 3u) >> 2)) * (64u * PACE_NUM / NCOMM);
        uint8_t* d = dbase + (size_t)(((c & 3u) * 32 + (c >> 2)) * 16384);
        const unsigned long long mask = ~0ull >> (64u - max(nl, 1u));
        while ((unsigned)(__builtin_amdgcn_s_memrealtime() - t0) < ((due + cost * b) >> 16)) __builtin_amdgcn_s_sleep(1);
        unit_send(x0, d, lane * 16, mask);
        if (i == 0) STAMP(4);
        if (i == 1) STAMP(5);
        if (i == n - 1) STAMP(6);
        due += cost * NCOMM;
        uint8_t* f = fbase + ((c & 3u) * 128 + (c >> 2)) * 8;
        const u32x2 fv = {call, meta};
        auto fetch = [&]() {
            asm volatile("" ::: "memory");
#pragma unroll
            for (int j = 0; j < 4; ++j) x0.g[j] = *(LAS u32x4*)(lds + (i + 1) * 4096 + j * 1024 + lane * 16);
        };
        meta = META_READY;
        if (i + 1 < n) { c = chunk(i + 1); meta = rfl(*(volatile LAS unsigned*)(metaw + i + 1)); if (meta & META_READY) fetch(); }
        waitv<0>();
        if (lane == 0) st8_sys(f, fv);
        if (!(meta & META_READY)) {
            do {
                __builtin_amdgcn_s_sleep(1);
                meta = rfl(*(volatile LAS unsigned*)(metaw + i + 1));
            } while (!(meta & META_READY));
            fetch();
        }
    }
}

struct Tile { int rb, cb; };
struct XRegs { u32x4 r[4]; unsigned e; };
DI void unit_load(XRegs& R, int ca, int cb, unsigned meta, const unsigned (&vx)[4], unsigned ve, __amdgpu_buffer_rsrc_t srd, unsigned so) {
    asm volatile("s_cmp_gt_i32 %13, %14\n\t"
                 "s_cbranch_scc0 2f\n\t"
                 "buffer_load_dwordx4 %0, %5, %10, %11 offen sc0 sc1\n\t"
                 "buffer_load_dwordx4 %1, %6, %10, %11 offen sc0 sc1\n\t"
                 "s_bitcmp1_b32 %12, 16\n\t"
                 "buffer_load_dwordx4 %2, %7, %10, %11 offen sc0 sc1\n\t"
                 "s_cbranch_scc0 1f\n\t"
                 "buffer_load_dwordx4 %3, %8, %10, %11 offen sc0 sc1\n\t"
                 "s_branch 2f\n"
                 "1:\n\t"
                 "buffer_load_sshort %4, %9, %10, %11 offen sc0 sc1\n"
                 "2:"
                 : "+v"(R.r[0]), "+v"(R.r[1]), "+v"(R.r[2]), "+v"(R.r[3]), "+v"(R.e)
                 : "v"(vx[0]), "v"(vx[1]), "v"(vx[2]), "v"(vx[3]), "v"(ve), "s"(srd), "s"(so), "s"(meta), "s"(ca), "s"(cb) : "memory", "scc");
}
DI void loader_role(const Params& p, const Tile tl, int li, int lane, LAS uint8_t* lds, Stamps& st) {
#define LOADER_PRIO 0
    __builtin_amdgcn_s_setprio(LOADER_PRIO);
    uint8_t* myws = p.myws;
    unsigned craw;
    {   unsigned long long keep;
        asm volatile("s_mov_b64 %1, exec\n\ts_mov_b64 exec, 1\n\tglobal_atomic_add %0, %2, %3, %4 sc0\n\ts_mov_b64 exec, %1\n\ts_waitcnt vmcnt(0)"
                     : "=&v"(craw), "=&s"(keep) : "v"((unsigned)(WS_CNT + (blockIdx.x * 8 + 4 + li) * 64)), "v"(1u), "s"(myws) : "memory");
    }
    const unsigned call = rfl(craw) + 1u;
#define WPF_STEP 3
#define WPF_FIRST 4
    {
        const unsigned g = blockIdx.x - NCOMM;
        for (unsigned i = 0; i < (tl.rb == 28 ? 0u : (g >> 3) + WPF_FIRST); ++i) __builtin_amdgcn_s_sleep(WPF_STEP);
        unsigned d;
        asm volatile("global_load_dword %0, %1, %2\n\ts_waitcnt vmcnt(0)" : "=&v"(d) : "v"((lane >> 4) * NFULL + (lane & 15) * 128), "s"(p.w + (size_t)(384u + (g * 4u + li) * 4u) * NFULL) : "memory");
    }
    const bool remote = tl.rb < 28;
    const int src = remote ? ((p.rank + 1 + (tl.rb % 7)) & 7) : p.rank, sub = remote ? tl.rb / 7 : 0;
    unsigned vx[4];
#pragma unroll
    for (int j = 0; j < 4; ++j)
        vx[j] = remote ? li * 4096 + j * 1024 + lane * 16 : ((li * 4 + j) * 8 + (lane >> 3)) * KDIM + (((lane & 7) ^ (4 * (j & 1) + (lane >> 4))) << 4);
    const unsigned vxe = li * 4096 + 3072 + lane * 2;
    const unsigned xk = remote ? 16384 : 128;
    const uint8_t* xb = remote ? myws + WS_DATA + (((size_t)(call & 1u) * NRANK + src) * 4 + sub) * (XBYTES / 4) : p.x;
    const __amdgpu_buffer_rsrc_t sX = mk_srd(xb, (unsigned)(XBYTES / 4));
    const uint8_t* fl = myws + WS_FLAG + (size_t)((call & 1u) * NRANK + src) * 4096 + ((sub * 4 + li) * 32 + (lane & 31)) * 8;
    asm volatile("s_nop 4" ::: "memory");

    int avail = remote ? 0 : 64; unsigned dead = 0;
    unsigned metav = META_RAW;
    u32x2 pv = {0u, 0u};
    XRegs R0 = {}, R1 = {};
    auto poll_async = [&](int lim) {
        asm volatile("s_cmp_gt_i32 %2, %3\n\ts_cbranch_scc1 1f\n\tglobal_load_dwordx2 %0, %1, off sc0 sc1\n1:" : "+v"(pv) : "v"(fl), "s"(avail), "s"(lim) : "memory", "scc");
    };
    auto take = [&](const u32x2& v) {
        const unsigned long long m = __builtin_amdgcn_ballot_w64(v.x < call);
        avail = m ? __builtin_ctzll(m) : 64;
        metav = v.y;
    };
    auto poll_sync = [&]() {
        u32x2 v;
        asm volatile("global_load_dwordx2 %0, %1, off sc0 sc1\n\ts_waitcnt vmcnt(0)" : "=&v"(v) : "v"(fl) : "memory");
        take(v);
    };
    unsigned m60 = 0x60606060u; asm("" : "+s"(m60));
    auto decode = [&](int s, const XRegs& R, unsigned meta, unsigned so) {
        LAS uint8_t* img = lds + so + li * 4096;
        if (__builtin_expect(meta & META_RAW, 0)) {
#pragma unroll
            for (int j = 0; j < 4; ++j) { *(LAS u32x4*)(img + j * 1024 + lane * 16) = R.r[j]; __builtin_amdgcn_s_sleep(1); }
        } else {
            const unsigned base4 = (meta & 0xffu) * 0x01010101u, tab = base4 + 0x90801000u;
            const unsigned nx = (meta >> 8) & 0xffu;
            auto patch = [&](unsigned e) {
                __hip_atomic_fetch_add((LAS unsigned*)(img + (e & 0xffcu)), (unsigned)((int)e >> 13) << (((e & 3u) << 3) + 5u), __ATOMIC_RELAXED, __HIP_MEMORY_SCOPE_WORKGROUP);
            };
            u32x4 sA, sB, sC;
            sA = R.r[0] & m60; asm("" : "+v"(sA));
            *(LAS u32x4*)(img + lane * 16) = (R.r[0] ^ sA) + base4; GAP();
            sB = R.r[1] & m60; asm("" : "+v"(sB));
            *(LAS u32x4*)(img + 1024 + lane * 16) = (R.r[1] ^ sB) + base4; GAP();
            sC = R.r[2] & m60; asm("" : "+v"(sC));
            *(LAS u32x4*)(img + 2048 + lane * 16) = (R.r[2] ^ sC) + base4; GAP();
            u32x4 o3;
            typedef unsigned long long u64;
#pragma unroll
            for (int d = 0; d < 4; d += 2) {
                u64 a = (((u64)sA[d + 1] << 32) | sA[d]) >> 5, b2 = (((u64)sB[d + 1] << 32) | sB[d]) >> 3, c = (((u64)sC[d + 1] << 32) | sC[d]) >> 5;
                asm("" : "+v"(a), "+v"(b2), "+v"(c));
                o3[d] = __builtin_amdgcn_perm(tab, tab, (unsigned)c) + (unsigned)a + (unsigned)b2;
                o3[d + 1] = __builtin_amdgcn_perm(tab, tab, (unsigned)(c >> 32)) + (unsigned)(a >> 32) + (unsigned)(b2 >> 32);
            }
            *(LAS u32x4*)(img + 3072 + lane * 16) = o3;
            asm volatile("" ::: "memory");
            patch((unsigned)lane < nx ? R.e : (unsigned)lane * 4u);
            if (__builtin_expect(nx > 64u, 0)) {
                unsigned e2;
                asm volatile("buffer_load_sshort %0, %1, %2, %3 offen offset:128 sc0 sc1\n\ts_waitcnt vmcnt(0)" : "=&v"(e2) : "v"(vxe), "s"(sX), "s"(s * xk) : "memory");
                patch((unsigned)lane + 64u < nx ? e2 : (unsigned)lane * 4u);
            }
        }
    };
    unsigned so = 0; int nl = 0;
    poll_async(1);
    auto step = [&](auto last, int s, XRegs& Rc, XRegs& Rn) {
        constexpr bool LAST = decltype(last)::value;
        asm volatile("s_cmp_gt_i32 %1, %2\n\ts_cbranch_scc1 1f\n\ts_waitcnt vmcnt(0)\n1:" : "+v"(pv) : "s"(avail), "s"(s + 1) : "memory", "scc");
        if (avail <= s + 1) {
            take(pv);
            if (avail <= s) {
                const unsigned long long t0 = __builtin_amdgcn_s_memrealtime();
                do {
                    poll_sync();
                    if (__builtin_amdgcn_s_memrealtime() - t0 > DEADLINE_TICKS && avail <= s) { dead = 1; avail = 64; metav = META_RAW; }
                } while (avail <= s);
            }
        }
        SEG(1);
        const unsigned meta = __builtin_amdgcn_readlane(metav, s);
        unit_load(Rc, s + 1, nl, meta, vx, vxe, sX, s * xk);
        if constexpr (!LAST) {
            if (nl == s && avail <= s + 1) poll_sync();
            unit_load(Rn, avail, s + 1, __builtin_amdgcn_readlane(metav, s + 1), vx, vxe, sX, (s + 1) * xk);
            nl = avail > s + 1 ? s + 2 : s + 1;
        }
        SEG(2);
        asm volatile("s_waitcnt vmcnt(%5)" : "+v"(Rc.r[0]), "+v"(Rc.r[1]), "+v"(Rc.r[2]), "+v"(Rc.r[3]), "+v"(Rc.e) : "i"(LAST ? 0 : 4) : "memory");
        if constexpr (!LAST) poll_async(s + 2);
        SEG(3);
        decode(s, Rc, meta, so);
        SEG(4);
        if constexpr (!LAST) {
            so = so == 2 * STG ? 0 : so + STG;
            asm volatile("s_waitcnt lgkmcnt(0)" ::: "memory");
            SEG(5);
            barrier();
            SEG(6);
        }
    };
    for (int s = 0; s < KSTEPS - 2; s += 2) { step(std::false_type{}, s, R0, R1); step(std::false_type{}, s + 1, R1, R0); }
    step(std::false_type{}, KSTEPS - 2, R0, R1);
    step(std::true_type{}, KSTEPS - 1, R1, R0);
    if (lane == 0) *(LAS unsigned*)(lds + LDS_DEAD + li * 4) = dead;
    asm volatile("s_waitcnt lgkmcnt(0)" ::: "memory");
    barrier();
    STAMP(5);
    STAMP_FLUSH();
}

DI v8i rd_x(LAS uint8_t* a0, LAS uint8_t* a1, int mt) {
    const v4i lo = *(LAS v4i*)(a0 + mt * 4096), hi = *(LAS v4i*)(a1 + mt * 4096);
    return __builtin_shufflevector(lo, hi, 0, 1, 2, 3, 4, 5, 6, 7);
}
DI v8i rd_w(LAS uint8_t* a, int kh) {
    const v2i r0 = __builtin_amdgcn_ds_read_tr8_b64_v2i32((LAS v2i*)(a + XS + kh * 16384));
    const v2i r1 = __builtin_amdgcn_ds_read_tr8_b64_v2i32((LAS v2i*)(a + XS + kh * 16384 + 2048));
    const v2i r2 = __builtin_amdgcn_ds_read_tr8_b64_v2i32((LAS v2i*)(a + XS + kh * 16384 + 4096));
    const v2i r3 = __builtin_amdgcn_ds_read_tr8_b64_v2i32((LAS v2i*)(a + XS + kh * 16384 + 6144));
    const v4i lo = __builtin_shufflevector(r0, r1, 0, 1, 2, 3), hi = __builtin_shufflevector(r2, r3, 0, 1, 2, 3);
    return __builtin_shufflevector(lo, hi, 0, 1, 2, 3, 4, 5, 6, 7);
}
#define MM(mt, nt, kh, xf) do { acc[mt][nt] = __builtin_amdgcn_mfma_scale_f32_32x32x64_f8f6f4(W[nt][kh], xf, acc[mt][nt], 1, 1, 0, 0, 0, 0); GAP(); } while (0)

template <int NM>
DI void mfma_role(const Params& p, const Tile tl, int w, int lane, LAS uint8_t* lds, Stamps& st) {
    const int r = lane & 31, g = lane >> 5, q = (lane & 15) >> 1;
    f32x16 acc[NM][2];
#pragma unroll
    for (int m = 0; m < NM; ++m) { acc[m][0] = (f32x16){}; acc[m][1] = (f32x16){}; }
    v8i W[2][2], xp0 = {}, xp1 = {};
    W[0][1] = (v8i){}; W[1][1] = (v8i){};
    const int xa = r * 128 + (((2 * g) ^ ((r >> 1) & 7)) << 4);
    const int wa = (32 * g + q) * 256 + (((4 * w + ((lane >> 4) & 1)) ^ (q << 1)) << 4) + 8 * (lane & 1);
    float sc = p.sx[0] * p.sw[0];
    asm volatile("" : "+v"(sc));
    unsigned row_main = tl.rb < 28 ? ((p.rank + 1 + (tl.rb % 7)) & 7) * MLOC + (tl.rb / 7) * 128 : p.rank * MLOC;
    unsigned row_extra = p.rank * MLOC + (tl.rb < NHEAVY ? 160 + 32 * tl.rb : 128);
    __amdgpu_buffer_rsrc_t sO = mk_srd(p.out, 4096u * NLOC * 4u);
    asm volatile("" : "+s"(row_main), "+s"(row_extra), "+s"(sO));
    const __amdgpu_buffer_rsrc_t sW = mk_srd(p.w + tl.cb * 256, (unsigned)(KDIM - 1) * NFULL + 256);
    const __amdgpu_buffer_rsrc_t sE = mk_srd(p.x + (size_t)(tl.rb < NHEAVY ? 160 + 32 * tl.rb : 128) * KDIM, 32 * KDIM);
    const unsigned vw0 = (32 * w + (lane >> 4)) * NFULL + (((lane & 15) ^ ((lane >> 4) << 1)) << 4), vw1 = vw0 ^ 128u;
    const unsigned ve = (8 * w + (lane >> 3)) * KDIM + (((lane & 7) ^ (4 * (w & 1) + (lane >> 4))) << 4);
    const unsigned ldsb = (unsigned)(uintptr_t)lds;
    auto piece = [&](auto ic, int t, unsigned sd) {
        constexpr int i = decltype(ic)::value;
        if constexpr (i == 8) { if constexpr (NM == 5) dma2<16384, 0, 0>(ldsb + sd + w * 1024, ve, sE, t * 128); }
        else dma2<XS + (i & 4) * 1024, i * 4 * NFULL - (i & 3) * 1024, (i & 3) * 1024>(ldsb + sd + w * 8192, (i & 1) ? vw1 : vw0, sW, t * (128 * NFULL));
        GAP();
    };
#define PIECE(i) piece(std::integral_constant<int, i>{}, td, sd)
    asm volatile("s_nop 4" ::: "memory");
    { int td = 0; unsigned sd = 0; PIECE(0); PIECE(1); PIECE(2); PIECE(3); PIECE(4); PIECE(5); PIECE(6); PIECE(7); PIECE(8); }
    { int td = 1; unsigned sd = STG; PIECE(0); PIECE(1); PIECE(2); PIECE(3); PIECE(4); PIECE(5); PIECE(6); PIECE(7); PIECE(8); }
    waitv<NM + 4>();
    int so = 0;
    auto kstep = [&](auto issue, int t) {
        constexpr bool ISSUE = decltype(issue)::value;
        LAS uint8_t* a00 = lds + (so + xa); LAS uint8_t* a01 = lds + (so + (xa ^ 16)); LAS uint8_t* a10 = lds + (so + (xa ^ 64)); LAS uint8_t* a11 = lds + (so + (xa ^ 80));
        LAS uint8_t* w0 = lds + (so + wa); LAS uint8_t* w1 = lds + (so + (wa ^ 32));
        asm volatile("" : "+v"(a00), "+v"(a01), "+v"(a10), "+v"(a11), "+v"(w0), "+v"(w1));
        const unsigned sd = so == 0 ? 2 * STG : so - STG; const int td = t + 2;
        so = (so == 2 * STG) ? 0 : so + STG;
        barrier(); GAP();
        v8i xf[2 * NM];
        MM(NM - 2, 0, 1, xp0);
        xf[0] = rd_x(a00, a01, 0); GAP();
        W[0][0] = rd_w(w0, 0); GAP();
        MM(NM - 2, 1, 1, xp0);
        W[1][0] = rd_w(w1, 0); GAP();
        MM(NM - 1, 0, 1, xp1);
        xf[1] = rd_x(a00, a01, 1); GAP();
        if constexpr (ISSUE) PIECE(0);
        MM(NM - 1, 1, 1, xp1);
#pragma unroll
        for (int f = 0; f < 2 * NM - 2; ++f) {
            const int fn = f + 2;
            xf[fn] = fn < NM ? rd_x(a00, a01, fn) : rd_x(a10, a11, fn - NM); GAP();
            if constexpr (ISSUE) { if (f == 0) PIECE(1); if (f == 1) PIECE(2); if (f == 2) PIECE(3); if (f == 3) PIECE(5); if (f == 4) PIECE(7); }
            MM(f % NM, 0, f / NM, xf[f]);
            if (f == 0) { W[0][1] = rd_w(w0, 1); GAP(); }
            if (f == 1) { W[1][1] = rd_w(w1, 1); GAP(); }
            if constexpr (ISSUE) { if (f == 2) PIECE(4); if (f == 3) PIECE(6); if (f == 4) PIECE(8); }
            MM(f % NM, 1, f / NM, xf[f]);
        }
        xp0 = xf[2 * NM - 2]; xp1 = xf[2 * NM - 1];
        asm volatile("s_waitcnt vmcnt(%2) lgkmcnt(0)" : "+v"(xp0), "+v"(xp1) : "i"(ISSUE ? NM + 4 : 0) : "memory"); GAP();
    };
    for (int t = 0; t < KSTEPS - 2; ++t) kstep(std::true_type{}, t);
    for (int t = KSTEPS - 2; t < KSTEPS; ++t) kstep(std::false_type{}, t);
    const u32x4 dd = *(LAS u32x4*)(lds + LDS_DEAD);
    MM(NM - 2, 0, 1, xp0); MM(NM - 2, 1, 1, xp0); MM(NM - 1, 0, 1, xp1); MM(NM - 1, 1, 1, xp1);
    STAMP(1);
    if (dd.x | dd.y | dd.z | dd.w) sc = __builtin_nanf("");
    LAS uint8_t* eb = lds + 2 * STG + w * 12288;
    const int ew = r * 128 + ((g ^ (r & 7)) << 4), er = (lane >> 3) * 128 + (((lane & 7) ^ (lane >> 3)) << 4);
    const unsigned vo = ((lane >> 3) * NLOC + tl.cb * 256 + w * 64 + 4 * (lane & 7)) * 4;
    auto stage = [&](int i, u32x4 (&v)[4]) {
        LAS uint8_t* e = eb + (i % 3) * 4096;
        const f32x16 a = acc[i >> 1][i & 1] * sc;
#pragma unroll
        for (int k = 0; k < 4; ++k) *(LAS f32x4*)(e + (ew ^ (k << 5))) = (f32x4){a[4 * k], a[4 * k + 1], a[4 * k + 2], a[4 * k + 3]};
#pragma unroll
        for (int k = 0; k < 4; ++k) v[k] = __builtin_bit_cast(u32x4, *(LAS f32x4*)(e + er + k * 1024));
    };
    auto put = [&](int i, u32x4 (&v)[4]) {
        const unsigned row = (i >> 1) < 4 ? row_main + 32 * (i >> 1) : row_extra;
#pragma unroll
        for (int k = 0; k < 4; ++k) __builtin_amdgcn_raw_buffer_store_b128(v[k], sO, vo, (row + 8 * k) * (NLOC * 4) + (i & 1) * 128, OUT_AUX);
    };
    u32x4 ev[2][4];
    stage(0, ev[0]);
#pragma unroll
    for (int i = 1; i < 2 * NM; ++i) { stage(i, ev[i & 1]); GAP(); put(i - 1, ev[(i - 1) & 1]); GAP(); }
    put(2 * NM - 1, ev[1]);
    STAMP(2);
    STAMP(3); STAMP_FLUSH();
}

__global__ __launch_bounds__(512, 2) void gemm_ag_kernel(const Params p) {
    extern __shared__ __attribute__((aligned(16))) uint8_t smem[];
    LAS uint8_t* lds = (LAS uint8_t*)smem;
    const int lane = threadIdx.x & 63, wave = rfl(threadIdx.x >> 6), b = blockIdx.x;
    Stamps st = {};
    STAMP(0);
    if (__builtin_expect(b < NCOMM, 1)) {
        comm_role(p, wave, lane, lds, st);
        return;
    }
    Tile tl;
    { const int xcd = b & 7, j = (b - NCOMM) >> 3;
      tl.rb = j; tl.cb = xcd;
    }
    if (wave == 0) {
        const unsigned j = (unsigned)(b - NCOMM) >> 3, k = j / 7u;
        if (k < 3u) {
            const uint8_t* t = p.peer[j - 7u * k] + (k == 2u ? WS_FLAG : WS_DATA + (size_t)(k * NRANK + p.rank) * XBYTES);
            unsigned d;
            asm volatile("global_load_dword %0, %1, %2 sc0 sc1\n\ts_waitcnt vmcnt(0)" : "=&v"(d) : "v"(0u), "s"(t) : "memory");
        }
    }
    if (wave >= 4) {
        loader_role(p, tl, wave - 4, lane, lds, st);
    } else {
        if (tl.rb < NHEAVY || tl.rb == 28) mfma_role<5>(p, tl, wave, lane, lds, st);
        else mfma_role<4>(p, tl, wave, lane, lds, st);
    }
}

}

extern "C" void kernel_launch(int rank, int nranks, ncclComm_t comm, hipStream_t stream,
                              void* const* d_in, const int* in_sizes, int n_in,
                              void* d_out, int out_size,
                              const int* in_dtypes, int out_dtype,
                              void* const* d_out_all,
                              void* const* d_ws_all, size_t ws_bytes) {
    if (nranks != NRANK || ws_bytes < WS_NEED || (unsigned)rank >= (unsigned)NRANK) { fprintf(stderr, "unexpected configuration\n"); return; }
    static bool attr_set = false;
    if (!attr_set) {
        HIP_CHECK(hipFuncSetAttribute((const void*)gemm_ag_kernel, hipFuncAttributeMaxDynamicSharedMemorySize, LDS_BYTES));
        attr_set = true;
    }
    Params p;
    p.x = (const uint8_t*)d_in[0]; p.w = (const uint8_t*)d_in[1] + (size_t)rank * NLOC; p.sx = (const float*)d_in[2]; p.sw = (const float*)d_in[3];
    p.out = (float*)d_out;
    p.myws = (uint8_t*)d_ws_all[rank];
    for (int j = 0; j < NRANK; ++j) p.peer[j] = (uint8_t*)d_ws_all[(rank + 1 + j) % NRANK];
    p.rank = rank; p.pad = 0;
    {
#define STAGV 0, 1, 2, 3
        const int stag[4] = {STAGV};
        uint8_t ord[128]; int n = 0;
        for (int key = 0; key < 2 * KSTEPS; ++key)
            for (int sub = 0; sub < 4; ++sub) { const int ks = key - stag[sub]; if (ks >= 0 && ks < KSTEPS) ord[n++] = (uint8_t)(sub | (ks << 2)); }
        for (int i = 0; i < 32; ++i) p.order[i] = ord[4 * i] | (ord[4 * i + 1] << 8) | (ord[4 * i + 2] << 16) | ((unsigned)ord[4 * i + 3] << 24);
    }
    hipLaunchKernelGGL(gemm_ag_kernel, dim3(GRID), dim3(512), LDS_BYTES, stream, p);
    HIP_CHECK(hipGetLastError());
}
```
